```python
import numpy as np

kernel_name = "gpu_gemm_ar_m4096_k4096_n16384_f32_reluq8"


ABSMAX_RTOL = 0.02

COMPUTE_DTYPE = "float32"
EPILOGUE = "relu_quant_int8"
IN_QUANT = None
OUT_QUANT = "int8"
IN_DTYPES = {"x": "float32", "w_mat": "float32"}
WORLD_SIZE = 8

def setup_inputs():

    x = np.random.randn(4096, 4096).astype(np.float32)
    w_mat = (np.random.randn(4096, 16384) / np.sqrt(4096)).astype(np.float32)
    return {"x": x, "w_mat": w_mat}

def _quant_dequant(y, amax):
    scale = amax / 127.0
    q = np.clip(np.round(y / scale), -127.0, 127.0)
    return (q * scale).astype(np.float32)
def _epilogue(y):
    y = np.maximum(y, 0.0)
    return _quant_dequant(y, np.abs(y).max())

def reference(x, w_mat):
    y = (x @ w_mat).astype(np.float32)
    return _epilogue(y)
```

```cpp
#include <hip/hip_runtime.h>
#include <rccl/rccl.h>
#include <cstdio>

#define LAS __attribute__((address_space(3)))
#define GAS __attribute__((address_space(1)))
typedef unsigned u32x4 __attribute__((ext_vector_type(4)));
typedef float f32x4 __attribute__((ext_vector_type(4)));
typedef float f32x2 __attribute__((ext_vector_type(2)));
typedef __bf16 bf16x2 __attribute__((ext_vector_type(2)));
typedef __bf16 bf16x8 __attribute__((ext_vector_type(8)));
typedef unsigned long long u64;

namespace {
constexpr int NR = 8;
constexpr int GM = 4096, GN = 16384, KSH = 512, NLOC = 2048;
constexpr int NWG = 256, NLINKWG = 56, NCONVWG = 72, GEMM0 = 128;
constexpr int WPL = 32;
constexpr int TASKS_IN = 1536;
constexpr int NCONV = 5120;
constexpr unsigned MiB = 1u << 20;
constexpr unsigned OFF_CALL = 0;
constexpr unsigned OFF_TMO = 16384;
constexpr unsigned OFF_AMAXL = 16384 + 256;
constexpr unsigned OFF_AMAXT = 16384 + 512;
constexpr unsigned OFF_AMAXG = 16384 + 1024;
constexpr unsigned OFF_CNTA = 32768;
constexpr unsigned OFF_CNTB = 32768 + 4096;
constexpr unsigned OFF_OFLAG = 49152;
constexpr unsigned OFF_HDR = 65536;
constexpr unsigned OFF_DBG = 524288;
constexpr unsigned OFF_STA = 1 * MiB;
constexpr unsigned OFF_STB = 33 * MiB;
constexpr unsigned OFF_EXB = 49 * MiB;
constexpr unsigned WS_NEED = 65 * MiB;
constexpr unsigned OUT_BYTES = (unsigned)GM * GN;
constexpr u64 DEADLINE = 50000000ull;
constexpr int LDS_BYTES = 131072 + 1024;
constexpr int AUX_SYS = 17, AUX_AG = 16;

struct Params {
  const float* x; const float* w;
  char* out[NR]; char* ws[NR];
  int rank;
};

__device__ __forceinline__ unsigned ld_sys(const char* p) { return __hip_atomic_load((const GAS unsigned*)p, __ATOMIC_RELAXED, __HIP_MEMORY_SCOPE_SYSTEM); }
__device__ __forceinline__ u64 ld_sys64(const char* p) { return __hip_atomic_load((const GAS u64*)p, __ATOMIC_RELAXED, __HIP_MEMORY_SCOPE_SYSTEM); }
__device__ __forceinline__ unsigned ld_ag(const char* p) { return __hip_atomic_load((const GAS unsigned*)p, __ATOMIC_RELAXED, __HIP_MEMORY_SCOPE_AGENT); }
__device__ __forceinline__ void st_sys(char* p, unsigned v) { __hip_atomic_store((GAS unsigned*)p, v, __ATOMIC_RELAXED, __HIP_MEMORY_SCOPE_SYSTEM); }
__device__ __forceinline__ void st_sys64(char* p, u64 v) { __hip_atomic_store((GAS u64*)p, v, __ATOMIC_RELAXED, __HIP_MEMORY_SCOPE_SYSTEM); }
__device__ __forceinline__ void st_ag(char* p, unsigned v) { __hip_atomic_store((GAS unsigned*)p, v, __ATOMIC_RELAXED, __HIP_MEMORY_SCOPE_AGENT); }
__device__ __forceinline__ void add_ag(char* p, unsigned v) { (void)__hip_atomic_fetch_add((GAS unsigned*)p, v, __ATOMIC_RELAXED, __HIP_MEMORY_SCOPE_AGENT); }
__device__ __forceinline__ int uni(int v) { return __builtin_amdgcn_readfirstlane(v); }
__device__ __forceinline__ __amdgpu_buffer_rsrc_t mk_rsrc(const void* p, unsigned bytes) { return __builtin_amdgcn_make_buffer_rsrc((void*)p, 0, (int)bytes, 0x00020000); }
__device__ __forceinline__ unsigned pk_bf16(float a, float b) { f32x2 v = {a, b}; return __builtin_bit_cast(unsigned, __builtin_convertvector(v, bf16x2)); }
#define WAIT_V0() asm volatile("s_waitcnt vmcnt(0)" ::: "memory")
#define NOW() __builtin_amdgcn_s_memrealtime()

__device__ __forceinline__ bool give_up(char* wsl, unsigned call, u64 t0) {
  if (ld_ag(wsl + OFF_TMO) == call || NOW() - t0 > DEADLINE) { st_ag(wsl + OFF_TMO, call); return true; }
  __builtin_amdgcn_s_sleep(2);
  return false;
}

#define STAMP(i) do {} while (0)

__device__ __forceinline__ void conv_task(const Params& P, char* wsl, int q, int lane, unsigned call) {
  const int rank = P.rank;
  const int t = q / 640, r = q % 640;
  const int kt = t * 8 + rank;
  const int c = lane & 15, kg = lane >> 4;
  __amdgpu_buffer_rsrc_t rl = mk_rsrc(wsl, WS_NEED);
  u32x4 f[4];
  if (r < 512) {
    const int dest = (rank + 1 + (r & 7)) & 7, j = (r >> 6), s = (r >> 3) & 7;
    const int g = s >> 1, ks = s & 1;
    const GAS float* src = (const GAS float*)P.w + (size_t)(t * 64 + ks * 32 + kg * 8) * GN + dest * NLOC + j * 256 + g * 64 + c * 4;
    f32x4 v[8];
#pragma unroll
    for (int i = 0; i < 8; ++i) v[i] = __builtin_nontemporal_load((const GAS f32x4*)(src + (size_t)i * GN));
#pragma unroll
    for (int n = 0; n < 4; ++n)
#pragma unroll
      for (int d = 0; d < 4; ++d) f[n][d] = pk_bf16(v[2 * d][n], v[2 * d + 1][n]);
    if (dest == rank) {
      const unsigned base = OFF_STB + (unsigned)(kt * 8 + j) * 32768u + s * 4096u;
#pragma unroll
      for (int n = 0; n < 4; ++n) __builtin_amdgcn_raw_buffer_store_b128(f[n], rl, lane * 16 + n * 1024, base, AUX_SYS);
      WAIT_V0();
      if (lane == 0) add_ag(wsl + OFF_CNTB + j * 256 + kt * 4, 1u);
    } else {
      const unsigned base = OFF_EXB + (unsigned)((dest * 8 + t) * 8 + j) * 32768u + s * 4096u;
#pragma unroll
      for (int n = 0; n < 4; ++n) __builtin_amdgcn_raw_buffer_store_b128(f[n], rl, lane * 16 + n * 1024, base, AUX_SYS);
      WAIT_V0();
      if (lane == 0) st_sys(P.ws[dest] + OFF_HDR + (rank * TASKS_IN + t * 192 + j * 8 + s) * 4, call);
    }
  } else {
    const int a = r - 512, i = a >> 3, s = a & 7;
    const GAS float* src = (const GAS float*)P.x + (size_t)(i * 256 + s * 32 + c) * KSH + t * 64 + kg * 8;
    f32x4 v[8];
#pragma unroll
    for (int rf = 0; rf < 2; ++rf)
#pragma unroll
      for (int ks = 0; ks < 2; ++ks)
#pragma unroll
        for (int h = 0; h < 2; ++h) v[(rf * 2 + ks) * 2 + h] = *(const GAS f32x4*)(src + rf * 16 * KSH + ks * 32 + h * 4);
#pragma unroll
    for (int n = 0; n < 4; ++n) {
      f[n][0] = pk_bf16(v[2 * n][0], v[2 * n][1]); f[n][1] = pk_bf16(v[2 * n][2], v[2 * n][3]);
      f[n][2] = pk_bf16(v[2 * n + 1][0], v[2 * n + 1][1]); f[n][3] = pk_bf16(v[2 * n + 1][2], v[2 * n + 1][3]);
    }
    const unsigned base = OFF_STA + (unsigned)(kt * 16 + i) * 32768u + s * 4096u;
#pragma unroll
    for (int n = 0; n < 4; ++n) __builtin_amdgcn_raw_buffer_store_b128(f[n], rl, lane * 16 + n * 1024, base, AUX_SYS);
    WAIT_V0();
    if (lane < NR) {
      if (lane == rank) add_ag(wsl + OFF_CNTA + i * 256 + kt * 4, 1u);
      else st_sys(P.ws[lane] + OFF_HDR + (rank * TASKS_IN + t * 192 + 64 + a) * 4, call);
    }
  }
}

__device__ __forceinline__ int link_slot(int n, int rel) {
  const int t = n / 192, r = n % 192, u = r >> 3, s = r & 7;
  const int uu = u < 8 ? ((u + rel) & 7) : 8 + ((u - 8 + (rel * 16) / 7) & 15);
  return t * 192 + uu * 8 + s;
}

__global__ __launch_bounds__(512, 2) void fused_kernel(Params P) {
  __shared__ __attribute__((aligned(1024))) char smem[LDS_BYTES];
  const int tid = threadIdx.x, lane = tid & 63, wid = uni(tid >> 6);
  const int b = blockIdx.x, rank = P.rank;
  char* wsl = P.ws[rank];
  STAMP(0);
  const unsigned call = (unsigned)uni((int)ld_ag(wsl + OFF_CALL + b * 64)) + 1u;

  if (b < NLINKWG) {
    if (wid < 4) {
      const int gw = b * 4 + wid, rel = gw % 7, k = gw / 7;
      const int src = (rank + 1 + rel) & 7;
      __amdgpu_buffer_rsrc_t rs = mk_rsrc(P.ws[src], WS_NEED), rl = mk_rsrc(wsl, WS_NEED);
      bool bad = false;
      {
        const int slot = link_slot(k + WPL * (lane < 48 ? lane : 0), rel);
        const int t = slot / 192, r = slot % 192, u = r >> 3, s = r & 7, kt = t * 8 + src;
        const unsigned soff = u < 8 ? OFF_EXB + (unsigned)((rank * 8 + t) * 8 + u) * 32768u + s * 4096u : OFF_STA + (unsigned)(kt * 16 + u - 8) * 32768u + s * 4096u;
        const unsigned doff = u < 8 ? OFF_STB + (unsigned)(kt * 8 + u) * 32768u + s * 4096u : OFF_STA + (unsigned)(kt * 16 + u - 8) * 32768u + s * 4096u;
        const unsigned coff = u < 8 ? OFF_CNTB + u * 256 + kt * 4 : OFF_CNTA + (u - 8) * 256 + kt * 4;
        const char* hp = wsl + OFF_HDR + (src * TASKS_IN + slot) * 4;
        u64 ready = 0;
        u32x4 v[4];
        for (int m = 0; m <= 48; ++m) {
          u32x4 o[4];
          WAIT_V0();
          if (m >= 2 && lane == m - 2) add_ag(wsl + coff, 1u);
          if (m >= 1) {
#pragma unroll
            for (int n = 0; n < 4; ++n) o[n] = v[n];
          }
          if (m < 48) {
            if (!((ready >> m) & 1)) {
              const u64 t0 = NOW();
              for (;;) {
                ready = __ballot(ld_sys(hp) == call);
                if ((ready >> m) & 1) break;
                if (give_up(wsl, call, t0)) { ready = ~0ull; bad = true; break; }
              }
            }
            const unsigned so = (unsigned)__builtin_amdgcn_readlane((int)soff, m);
#pragma unroll
            for (int n = 0; n < 4; ++n) v[n] = __builtin_amdgcn_raw_buffer_load_b128(rs, lane * 16 + n * 1024, so, AUX_SYS);
          }
          if (m >= 1) {
            const unsigned dof = (unsigned)__builtin_amdgcn_readlane((int)doff, m - 1);
#pragma unroll
            for (int n = 0; n < 4; ++n) __builtin_amdgcn_raw_buffer_store_b128(o[n], rl, lane * 16 + n * 1024, dof, AUX_AG);
          }
          if (m == 0) STAMP(1);
        }
        WAIT_V0();
        if (lane == 47) add_ag(wsl + coff, 1u);
      }
      STAMP(2);
      {
        __amdgpu_buffer_rsrc_t ro = mk_rsrc(P.out[src], OUT_BYTES), rd = mk_rsrc(P.out[rank], OUT_BYTES);
        const char* fp = wsl + OFF_OFLAG + (src * 128 + 2 * lane + (k >> 4)) * 4;
        const unsigned voff = (unsigned)(lane >> 4) * GN + (lane & 15) * 16;
        u64 ready = 0;
        u32x4 v[4]; unsigned prev = 0;
        for (int m = 0; m <= 64; ++m) {
          u32x4 o[4];
          WAIT_V0();
          if (m >= 1) {
#pragma unroll
            for (int n = 0; n < 4; ++n) o[n] = v[n];
          }
          const int tile = 2 * m + (k >> 4);
          const unsigned so = (unsigned)((tile >> 3) * 256 + (k & 15) * 16) * GN + src * NLOC + (tile & 7) * 256;
          if (m < 64) {
            if (!((ready >> m) & 1)) {
              const u64 t0 = NOW();
              for (;;) {
                ready = __ballot(ld_sys(fp) == call);
                if ((ready >> m) & 1) break;
                if (give_up(wsl, call, t0)) { ready = ~0ull; bad = true; break; }
              }
              if (m == 0) STAMP(3);
            }
#pragma unroll
            for (int n = 0; n < 4; ++n) v[n] = __builtin_amdgcn_raw_buffer_load_b128(ro, voff + n * 4 * GN, so, AUX_SYS);
          }
          if (m >= 1) {
            if (bad) { const u32x4 z = {0x80808080u, 0x80808080u, 0x80808080u, 0x80808080u}; o[0] = z; }
#pragma unroll
            for (int n = 0; n < 4; ++n) __builtin_amdgcn_raw_buffer_store_b128(o[n], rd, voff + n * 4 * GN, prev, 0);
          }
          prev = so;
        }
      }
      STAMP(4);
    }
  } else if (b < GEMM0) {
    const int cw = (b - NLINKWG) * 8 + wid;
    for (int q = 1024 + cw; q < NCONV; q += NCONVWG * 8) conv_task(P, wsl, q, lane, call);
    STAMP(1);
  } else {
    const int g = b - GEMM0;
    conv_task(P, wsl, g * 8 + wid, lane, call);
    STAMP(1);
    const int xcd = g & 7, idx = g >> 3;
    const int pm = (xcd >> 1) * 4 + (idx >> 2), pn = (xcd & 1) * 4 + (idx & 3);
    const int wr = wid >> 2, wc = wid & 3, fr = lane & 15, fq = lane >> 4;
    LAS char* lds = (LAS char*)smem;
    volatile LAS u64* ctl = (volatile LAS u64*)(lds + 131072);
    __amdgpu_buffer_rsrc_t rl = mk_rsrc(wsl, WS_NEED);
    const unsigned want = call * 8u;
    const char* pA = wsl + OFF_CNTA + pm * 256 + lane * 4;
    const char* pB = wsl + OFF_CNTB + pn * 256 + lane * 4;
    f32x4 acc[8][4];
#pragma unroll
    for (int m = 0; m < 8; ++m)
#pragma unroll
      for (int n = 0; n < 4; ++n) acc[m][n] = (f32x4){0.f, 0.f, 0.f, 0.f};
    const unsigned gvoff = wid * 4096 + lane * 16;
#define STAGE(buf, kt) do { \
      const unsigned sa_ = OFF_STA + (unsigned)((kt) * 16 + pm) * 32768u, sb_ = OFF_STB + (unsigned)((kt) * 8 + pn) * 32768u; \
      LAS char* d_ = lds + (buf) * 65536 + wid * 4096; \
      __builtin_amdgcn_raw_ptr_buffer_load_lds(rl, d_, 16, gvoff, sa_, 0, 0); \
      __builtin_amdgcn_raw_ptr_buffer_load_lds(rl, d_, 16, gvoff, sa_, 1024, 0); \
      __builtin_amdgcn_raw_ptr_buffer_load_lds(rl, d_, 16, gvoff, sa_, 2048, 0); \
      __builtin_amdgcn_raw_ptr_buffer_load_lds(rl, d_, 16, gvoff, sa_, 3072, 0); \
      __builtin_amdgcn_raw_ptr_buffer_load_lds(rl, d_ + 32768, 16, gvoff, sb_, 0, 0); \
      __builtin_amdgcn_raw_ptr_buffer_load_lds(rl, d_ + 32768, 16, gvoff, sb_, 1024, 0); \
      __builtin_amdgcn_raw_ptr_buffer_load_lds(rl, d_ + 32768, 16, gvoff, sb_, 2048, 0); \
      __builtin_amdgcn_raw_ptr_buffer_load_lds(rl, d_ + 32768, 16, gvoff, sb_, 3072, 0); } while (0)
#define BAR() do { asm volatile("s_waitcnt lgkmcnt(0)" ::: "memory"); __builtin_amdgcn_s_barrier(); asm volatile("" ::: "memory"); } while (0)
    u64 issued = 0, ready = 0;
    bool pre = false;
    const int aoff = wr * 16384 + lane * 16, boff = 32768 + wc * 8192 + lane * 16;
    for (int it = 0; it < 64; ++it) {
      if (!pre) {
        if ((ready & ~issued) == 0) {
          if (wid == 0) {
            const u64 t0 = NOW(); u64 rdy;
            for (;;) {
              rdy = __ballot(ld_ag(pA) >= want && ld_ag(pB) >= want);
              if (rdy & ~issued) break;
              if (give_up(wsl, call, t0)) { rdy = ~0ull; break; }
            }
            if (lane == 0) ctl[2] = rdy;
          }
          BAR();
          const u64 rv = ctl[2];
          ready = ((u64)(unsigned)uni((int)(rv >> 32)) << 32) | (unsigned)uni((int)rv);
        }
        const int kt = __builtin_ctzll(ready & ~issued); issued |= 1ull << kt;
        STAGE(it & 1, kt);
        WAIT_V0(); BAR();
      }
      const u64 avail = ready & ~issued;
      pre = avail != 0;
      if (pre) { const int kt = __builtin_ctzll(avail); issued |= 1ull << kt; STAGE((it + 1) & 1, kt); }
      unsigned pa = 0, pb = 0;
      if (wid == 0) asm volatile("global_load_dword %0, %2, off sc1\n\tglobal_load_dword %1, %3, off sc1" : "=&v"(pa), "=&v"(pb) : "v"(pA), "v"(pB) : "memory");
      {
        const LAS char* bb = lds + (it & 1) * 65536;
#pragma unroll
        for (int ks = 0; ks < 2; ++ks) {
          bf16x8 af[8], bf[4];
#pragma unroll
          for (int n = 0; n < 4; ++n) bf[n] = *(const LAS bf16x8*)(bb + boff + ks * 4096 + n * 1024);
#pragma unroll
          for (int m = 0; m < 8; ++m) af[m] = *(const LAS bf16x8*)(bb + aoff + m * 2048 + ks * 1024);
#pragma unroll
          for (int m = 0; m < 8; ++m)
#pragma unroll
            for (int n = 0; n < 4; ++n) acc[m][n] = __builtin_amdgcn_mfma_f32_16x16x32_bf16(bf[n], af[m], acc[m][n], 0, 0, 0);
          __builtin_amdgcn_sched_barrier(0);
        }
      }
      asm volatile("s_waitcnt vmcnt(0)" : "+v"(pa), "+v"(pb) :: "memory");
      if (wid == 0) { const u64 rdy = __ballot(pa >= want && pb >= want) | ready; if (lane == 0) ctl[it & 1] = rdy; }
      BAR();
      const u64 rv = ctl[it & 1];
      ready = ((u64)(unsigned)uni((int)(rv >> 32)) << 32) | (unsigned)uni((int)rv);
    }
    STAMP(2);
    float mx = 0.f;
#pragma unroll
    for (int m = 0; m < 8; ++m)
#pragma unroll
      for (int n = 0; n < 4; ++n) mx = fmaxf(fmaxf(mx, fmaxf(acc[m][n][0], acc[m][n][1])), fmaxf(acc[m][n][2], acc[m][n][3]));
#pragma unroll
    for (int o = 32; o >= 1; o >>= 1) mx = fmaxf(mx, __shfl_xor(mx, o));
    volatile LAS float* red = (volatile LAS float*)(lds + 131072 + 64);
    if (lane == 0) red[wid] = mx;
    BAR();
    if (wid == 0) {
      float v = red[lane & 7];
#pragma unroll
      for (int o = 4; o >= 1; o >>= 1) v = fmaxf(v, __shfl_xor(v, o));
      unsigned last = 0;
      if (lane == 0) {
        (void)__hip_atomic_fetch_max((GAS u64*)(wsl + OFF_AMAXL), ((u64)call << 32) | __float_as_uint(v), __ATOMIC_RELAXED, __HIP_MEMORY_SCOPE_AGENT);
        WAIT_V0();
        last = __hip_atomic_fetch_add((GAS unsigned*)(wsl + OFF_AMAXT), 1u, __ATOMIC_RELAXED, __HIP_MEMORY_SCOPE_AGENT) + 1u == call * 128u;
      }
      if (uni((int)last)) {
        const u64 gm = __hip_atomic_load((GAS u64*)(wsl + OFF_AMAXL), __ATOMIC_RELAXED, __HIP_MEMORY_SCOPE_AGENT);
        if (lane < NR) st_sys64(P.ws[lane] + OFF_AMAXG + rank * 128, gm);
      }
      const u64 t0 = NOW(); u64 gv;
      for (;;) {
        gv = ld_sys64(wsl + OFF_AMAXG + (lane & 7) * 128);
        if (__all((unsigned)(gv >> 32) == call)) break;
        if (give_up(wsl, call, t0)) break;
      }
      float am = __uint_as_float((unsigned)gv);
#pragma unroll
      for (int o = 4; o >= 1; o >>= 1) am = fmaxf(am, __shfl_xor(am, o));
      const unsigned tm = ld_ag(wsl + OFF_TMO);
      if (lane == 0) { red[8] = am; red[9] = (tm == call) ? 1.f : 0.f; }
    }
    BAR();
    STAMP(3);
    const float amax = red[8]; const bool bad = red[9] != 0.f;
    const float inv = amax > 0.f ? 127.f / amax : 0.f;
    __amdgpu_buffer_rsrc_t rd = mk_rsrc(P.out[rank], OUT_BYTES);
    const unsigned ovoff = (unsigned)(wr * 128 + fr) * GN + wc * 64 + fq * 16;
    const unsigned osoff = (unsigned)(pm * 256) * GN + rank * NLOC + pn * 256;
#pragma unroll
    for (int m = 0; m < 8; ++m) {
      u32x4 q;
#pragma unroll
      for (int r = 0; r < 4; ++r) {
        unsigned d = 0;
#pragma unroll
        for (int n = 0; n < 4; ++n) d = __builtin_amdgcn_cvt_pk_u8_f32(__builtin_rintf(acc[m][n][r] * inv), n, d);
        q[r] = bad ? 0x80808080u : d;
      }
      __builtin_amdgcn_raw_buffer_store_b128(q, rd, ovoff + m * 16 * GN, osoff, AUX_SYS);
    }
    WAIT_V0(); BAR();
    if (tid < NR && tid != rank) st_sys(P.ws[tid] + OFF_OFLAG + (rank * 128 + pm * 8 + pn) * 4, call);
    STAMP(4);
  }
  __syncthreads();
  if (tid == 0) st_ag(wsl + OFF_CALL + b * 64, call);
}
}

extern "C" void kernel_launch(int rank, int nranks, ncclComm_t comm, hipStream_t stream,
                              void* const* d_in, const int* in_sizes, int n_in,
                              void* d_out, int out_size, const int* in_dtypes, int out_dtype,
                              void* const* d_out_all, void* const* d_ws_all, size_t ws_bytes) {
  if (nranks != NR || (unsigned)rank >= (unsigned)NR || ws_bytes < WS_NEED || out_size != GM * GN) { fprintf(stderr, "unexpected configuration\n"); return; }
  Params P;
  P.x = (const float*)d_in[0]; P.w = (const float*)d_in[1];
  for (int p = 0; p < NR; ++p) { P.out[p] = (char*)d_out_all[p]; P.ws[p] = (char*)d_ws_all[p]; }
  P.rank = rank;
  hipLaunchKernelGGL(fused_kernel, dim3(NWG), dim3(512), 0, stream, P);
}
```

```cpp
#include <hip/hip_runtime.h>
#include <rccl/rccl.h>
#include <cstdio>

#define LAS __attribute__((address_space(3)))
#define GAS __attribute__((address_space(1)))
typedef unsigned u32x4 __attribute__((ext_vector_type(4)));
typedef unsigned u32x3 __attribute__((ext_vector_type(3)));
typedef unsigned u32x2 __attribute__((ext_vector_type(2)));
typedef u32x2 u32x2a4 __attribute__((aligned(4)));
typedef u32x3 u32x3a4 __attribute__((aligned(4)));
typedef float f32x4 __attribute__((ext_vector_type(4)));
typedef float f32x2 __attribute__((ext_vector_type(2)));
typedef __bf16 bf16x2 __attribute__((ext_vector_type(2)));
typedef __bf16 bf16x8 __attribute__((ext_vector_type(8)));
typedef unsigned long long u64;

namespace {
constexpr int NR = 8;
constexpr int GM = 4096, GN = 16384, KSH = 512, NLOC = 2048;
constexpr int NWG = 256, NLINKWG = 56, NCONVWG = 24, GEMM0 = 80, NGEMM = 176;
constexpr int NSUB = 1376;
constexpr int WPL = 32;
constexpr int DPL = 44;
constexpr int TASKS_IN = 1536;
constexpr int NUNIT = 640;
constexpr unsigned MiB = 1u << 20;
constexpr unsigned OFF_CALL = 0;
constexpr unsigned OFF_TMO = 16384;
constexpr unsigned OFF_AMAXL = 16384 + 256;
constexpr unsigned OFF_AMAXT = 16384 + 512;
constexpr unsigned OFF_AMAXG = 16384 + 1024;
constexpr unsigned CSTR = 4352;
constexpr unsigned OFF_CNTA = 131072;
constexpr unsigned OFF_CNTB = OFF_CNTA + 22 * CSTR;
constexpr unsigned OFF_TICK = OFF_CNTB + 8 * CSTR;
constexpr unsigned OFF_MFLAG = 49152;
constexpr unsigned OFF_VFLAG = 49152 + 8192;
constexpr unsigned OFF_UFLAG = 65536;
constexpr unsigned OFF_DBG = 524288;
constexpr unsigned OFF_STA = 1 * MiB;
constexpr unsigned OFF_STB = 33 * MiB;
constexpr unsigned OFF_XLE = 49 * MiB;
constexpr unsigned OFF_XH = 64 * MiB;
constexpr unsigned OFF_XX = 74 * MiB;
constexpr unsigned OFF_EXM = 75 * MiB;
constexpr unsigned OFF_EXV = 77 * MiB;
constexpr unsigned OFF_EXD = 84 * MiB;
constexpr unsigned OFF_EXH = 86 * MiB;
constexpr unsigned WS_NEED = 87 * MiB;
constexpr unsigned OUT_BYTES = (unsigned)GM * GN;
constexpr u64 DEADLINE = 50000000ull;
constexpr int LDS_CTL = 131072;
constexpr int LDS_BYTES = 131072 + 2048;
constexpr int AUX_SYS = 17, AUX_AG = 16;
#define AUX_OUT 0

struct Params {
  const float* x; const float* w;
  char* out; char* ws[NR];
  int rank;
};

__device__ __forceinline__ unsigned ld_sys(const char* p) { return __hip_atomic_load((const GAS unsigned*)p, __ATOMIC_RELAXED, __HIP_MEMORY_SCOPE_SYSTEM); }
__device__ __forceinline__ u64 ld_sys64(const char* p) { return __hip_atomic_load((const GAS u64*)p, __ATOMIC_RELAXED, __HIP_MEMORY_SCOPE_SYSTEM); }
__device__ __forceinline__ unsigned ld_ag(const char* p) { return __hip_atomic_load((const GAS unsigned*)p, __ATOMIC_RELAXED, __HIP_MEMORY_SCOPE_AGENT); }
__device__ __forceinline__ void st_sys(char* p, unsigned v) { __hip_atomic_store((GAS unsigned*)p, v, __ATOMIC_RELAXED, __HIP_MEMORY_SCOPE_SYSTEM); }
__device__ __forceinline__ void st_sys64(char* p, u64 v) { __hip_atomic_store((GAS u64*)p, v, __ATOMIC_RELAXED, __HIP_MEMORY_SCOPE_SYSTEM); }
__device__ __forceinline__ void st_ag(char* p, unsigned v) { __hip_atomic_store((GAS unsigned*)p, v, __ATOMIC_RELAXED, __HIP_MEMORY_SCOPE_AGENT); }
__device__ __forceinline__ void add_ag(char* p, unsigned v) { (void)__hip_atomic_fetch_add((GAS unsigned*)p, v, __ATOMIC_RELAXED, __HIP_MEMORY_SCOPE_AGENT); }
__device__ __forceinline__ int uni(int v) { return __builtin_amdgcn_readfirstlane(v); }
__device__ __forceinline__ __amdgpu_buffer_rsrc_t mk_rsrc(const void* p, unsigned bytes) { return __builtin_amdgcn_make_buffer_rsrc((void*)p, 0, (int)bytes, 0x00020000); }
__device__ __forceinline__ unsigned pk_bf16(float a, float b) { f32x2 v = {a, b}; return __builtin_bit_cast(unsigned, __builtin_convertvector(v, bf16x2)); }
#define WAIT_V(n) asm volatile("s_waitcnt vmcnt(" #n ")" ::: "memory")
#define WAIT_V0() do { WAIT_V(0); __builtin_amdgcn_s_waitcnt(0x0F70); } while (0)
#define FENCE() asm volatile("" ::: "memory")
#define BAR() do { asm volatile("" ::: "memory"); __builtin_amdgcn_s_waitcnt(0xC07F); __builtin_amdgcn_s_barrier(); asm volatile("" ::: "memory"); } while (0)
#define SCHED() __builtin_amdgcn_sched_barrier(0)
#define NOW() __builtin_amdgcn_s_memrealtime()

__device__ __forceinline__ bool give_up(char* wsl, unsigned call, u64 t0) {
  const u64 el = NOW() - t0;
  if (el > 100000ull && (el > DEADLINE || ld_ag(wsl + OFF_TMO) == call)) { st_ag(wsl + OFF_TMO, call); return true; }
  __builtin_amdgcn_s_sleep(2);
  return false;
}

#define STAMP(i) do {} while (0)
#define STAMPV(i, v) do {} while (0)
#define DBG(x)

__device__ __forceinline__ unsigned scan_incl(unsigned v) {
  v += __builtin_amdgcn_update_dpp(0, (int)v, 0x111, 0xf, 0xf, false);
  v += __builtin_amdgcn_update_dpp(0, (int)v, 0x112, 0xf, 0xf, false);
  v += __builtin_amdgcn_update_dpp(0, (int)v, 0x114, 0xf, 0xf, false);
  v += __builtin_amdgcn_update_dpp(0, (int)v, 0x118, 0xf, 0xf, false);
  v += __builtin_amdgcn_update_dpp(0, (int)v, 0x142, 0xa, 0xf, false);
  v += __builtin_amdgcn_update_dpp(0, (int)v, 0x143, 0xc, 0xf, false);
  return v;
}
__device__ __forceinline__ unsigned piece_offsets(const u32x3 mk, unsigned (&off)[6]) {
  unsigned base = 0;
#pragma unroll
  for (int D = 0; D < 3; ++D) {
    const unsigned c = __builtin_popcount(mk[D] & 0x0f0f0f0fu) | (__builtin_popcount(mk[D] & 0xf0f0f0f0u) << 16);
    const unsigned s = scan_incl(c), ex = s - c, tot = (unsigned)__builtin_amdgcn_readlane((int)s, 63);
    off[2 * D] = base + (ex & 0xffffu); base += tot & 0xffffu;
    off[2 * D + 1] = base + (ex >> 16); base += tot >> 16;
  }
  return base;
}

__device__ __forceinline__ void conv_unit(const Params& P, char* wsl, LAS char* lds, int U, int wid, int lane, unsigned prev) {
  const int rank = P.rank;
  const int t = U / 80, r = U % 80, s = wid;
  const int kt = t * 8 + rank;
  const int c = lane & 15, kg = lane >> 4;
  const bool isB = r < 64;
  const int dest = (rank + 1 + (r & 7)) & 7, j = r >> 3, i = r - 64;
  __amdgpu_buffer_rsrc_t rl = mk_rsrc(wsl, WS_NEED);
  unsigned f[16];
  if (isB) {
    const GAS float* src = (const GAS float*)P.w + (size_t)(t * 64 + (s >> 2) * 32 + kg * 8) * GN + dest * NLOC + j * 256 + (s & 3) * 64 + c * 4;
    f32x4 v[8];
#pragma unroll
    for (int q = 0; q < 8; ++q) v[q] = __builtin_nontemporal_load((const GAS f32x4*)(src + (size_t)q * GN));
#pragma unroll
    for (int n = 0; n < 4; ++n)
#pragma unroll
      for (int d = 0; d < 4; ++d) f[n * 4 + d] = pk_bf16(v[2 * d][n], v[2 * d + 1][n]);
  } else {
    const GAS float* src = (const GAS float*)P.x + (size_t)(i * 256 + (s & 3) * 64 + c) * KSH + t * 64 + (s >> 2) * 32 + kg * 8;
    f32x4 v[8];
#pragma unroll
    for (int q = 0; q < 8; ++q) v[q] = *(const GAS f32x4*)(src + (q >> 1) * 16 * KSH + (q & 1) * 4);
#pragma unroll
    for (int q = 0; q < 8; ++q) { f[2 * q] = pk_bf16(v[q][0], v[q][1]); f[2 * q + 1] = pk_bf16(v[q][2], v[q][3]); }
  }
  const bool local = !isB || dest == rank, remote = !isB || dest != rank;
  if (remote) {
    const unsigned e = (unsigned)((t * 80 + (isB ? dest * 8 + j : r)) * 8 + s);
    u32x4 L0, L1, E; unsigned H[8];
    unsigned mxv = 0;
#pragma unroll
    for (int q = 0; q < 8; ++q) {
      const unsigned lo = __builtin_amdgcn_perm(f[2 * q + 1], f[2 * q], 0x06040200u);
      H[q] = __builtin_amdgcn_perm(f[2 * q + 1], f[2 * q], 0x07050301u);
      if (q < 4) L0[q] = lo; else L1[q - 4] = lo;
      const unsigned a = f[2 * q] & 0x7fff7fffu, bq = f[2 * q + 1] & 0x7fff7fffu;
      mxv = max(mxv, max(max(a, a << 16), max(bq, bq << 16)));
    }
    __builtin_amdgcn_raw_buffer_store_b128(L0, rl, lane * 16, OFF_XLE + e * 3072u, AUX_SYS);
    __builtin_amdgcn_raw_buffer_store_b128(L1, rl, lane * 16 + 1024, OFF_XLE + e * 3072u, AUX_SYS);
#pragma unroll
    for (int o = 32; o >= 1; o >>= 1) mxv = max(mxv, (unsigned)__shfl_xor((int)mxv, o));
    unsigned ref = (unsigned)uni((int)(mxv >> 24));
    const unsigned R = ref * 0x01010101u;
    unsigned any = 0;
#pragma unroll
    for (int q = 0; q < 4; ++q) {
      const unsigned d0 = R - (H[2 * q] & 0x7f7f7f7fu), d1 = R - (H[2 * q + 1] & 0x7f7f7f7fu);
      any |= d0 | d1;
      E[q] = (d0 & 0x07070707u) | ((H[2 * q] >> 4) & 0x08080808u) | ((d1 & 0x07070707u) << 4) | (H[2 * q + 1] & 0x80808080u);
    }
    unsigned nexc = 0;
    if (__ballot((any & 0x78787878u) != 0u)) {
      LAS unsigned* hd = (LAS unsigned*)(lds + LDS_CTL + 256 + wid * 128);
#pragma unroll
      for (int q = 0; q < 8; ++q) {
        const unsigned d = R - (H[q] & 0x7f7f7f7fu);
        unsigned ov = (((d & 0x78787878u) + 0x78787878u) & 0x80808080u);
        for (;;) {
          const u64 bal = __ballot(ov != 0u);
          if (!bal || nexc > 15u) break;
          const unsigned sl = nexc + __builtin_amdgcn_mbcnt_hi((unsigned)(bal >> 32), __builtin_amdgcn_mbcnt_lo((unsigned)bal, 0u));
          if (ov != 0u && sl < 15u) { const unsigned sh = __builtin_ctz(ov) & 24u; hd[1 + sl] = (unsigned)lane | (q << 8) | (sh << 16) | (((H[q] >> sh) & 0xffu) << 24); }
          ov &= ov - 1;
          nexc += __builtin_popcountll(bal);
        }
      }
      if (nexc <= 15u) {
        if (lane == 0) hd[0] = nexc;
        if (lane < 16) __builtin_amdgcn_raw_buffer_store_b32(hd[lane], rl, lane * 4, OFF_XX + e * 64u, AUX_SYS);
      }
    }
    if (nexc > 15u || ref == 127u) {
      ref = 127u; nexc = 0;
      const u32x4 H0 = {H[0], H[1], H[2], H[3]}, H1 = {H[4], H[5], H[6], H[7]};
      __builtin_amdgcn_raw_buffer_store_b128(H0, rl, lane * 16, OFF_XH + e * 2048u, AUX_SYS);
      __builtin_amdgcn_raw_buffer_store_b128(H1, rl, lane * 16 + 1024, OFF_XH + e * 2048u, AUX_SYS);
    } else {
      __builtin_amdgcn_raw_buffer_store_b128(E, rl, lane * 16 + 2048, OFF_XLE + e * 3072u, AUX_SYS);
    }
    if (lane == 0) *(volatile LAS unsigned char*)(lds + LDS_CTL + 128 + wid) = (unsigned char)(ref | (nexc ? 0x80u : 0u));
  }
  if (local) {
    const unsigned base = isB ? OFF_STB + (unsigned)(kt * 8 + j) * 32768u + s * 4096u : OFF_STA + (unsigned)((kt * 2 + (s >> 2)) * 64 + i * 4 + (s & 3)) * 4096u;
#pragma unroll
    for (int n = 0; n < 4; ++n) { const u32x4 o = {f[4 * n], f[4 * n + 1], f[4 * n + 2], f[4 * n + 3]}; __builtin_amdgcn_raw_buffer_store_b128(o, rl, lane * 16 + n * 1024, base, AUX_AG); }
  }
  WAIT_V0();
  if (local && lane == 0) add_ag(wsl + (isB ? OFF_CNTB + j * CSTR : OFF_CNTA + ((i * 4 + (s & 3)) / 3) * CSTR) + kt * 4, 1u);
  BAR();
  if (wid == 0) {
    if (remote && lane < 2 * NR) {
      const int pd = lane >> 1, hf = lane & 1;
      const unsigned pay = *(volatile LAS unsigned*)(lds + LDS_CTL + 128 + hf * 4);
      if (pd != rank && (!isB || pd == dest))
        st_sys64(P.ws[pd] + OFF_UFLAG + rank * CSTR + (t * 24 + (isB ? j : 8 + i)) * 16 + hf * 8, ((u64)pay << 32) | (prev + 1u));
    }
  }
  BAR();
}

__global__ __launch_bounds__(512, 2) void fused_kernel(Params P) {
  __shared__ __attribute__((aligned(1024))) char smem[LDS_BYTES];
  const int tid = threadIdx.x, lane = tid & 63, wid = uni(tid >> 6);
  const int b = blockIdx.x, rank = P.rank;
  char* wsl = P.ws[rank];
  LAS char* lds = (LAS char*)smem;
  STAMP(0);
  DBG(const u64 tstart = NOW();)
  const unsigned prev = ld_ag(wsl + OFF_CALL + b * 64);

  if (b < GEMM0) {
    bool bad = false;
    if (b < NLINKWG) {
      if (wid < 4) {
        const unsigned call = (unsigned)uni((int)prev) + 1u;
        const int gw = b * 4 + wid, rel = gw % 7, k = gw / 7;
        const int src = (rank + 1 + rel) & 7;
        __amdgpu_buffer_rsrc_t rs = mk_rsrc(P.ws[src], WS_NEED), rl = mk_rsrc(wsl, WS_NEED);
        const unsigned tbase = (call - 1u) * (unsigned)TASKS_IN - (unsigned)WPL;
        char* tp = wsl + OFF_TICK + rel * CSTR;
        const unsigned fo = OFF_UFLAG + src * CSTR;
        u64 rdy0 = 0, rdy1 = 0, rdy2 = 0;
        u32x4 fl0 = {0, 0, 0, 0}, fl1 = fl0, fl2 = fl0;
        u32x4 v[4]; unsigned tk = tbase + (unsigned)k, hx = 0;
        char* wsr = P.ws[src];
        unsigned cur_d = 0, cur_c = 0, cur_m = 0, prv_c = 0; bool cur = false, prv = false, more = true;
        const u64 t0 = NOW();
#define FLAGS_LOAD() do { \
          if (~rdy0) fl0 = __builtin_amdgcn_raw_buffer_load_b128(rl, lane * 16, fo, AUX_SYS); \
          if (~rdy1 && U >= 24) fl1 = __builtin_amdgcn_raw_buffer_load_b128(rl, lane * 16 + 1024, fo, AUX_SYS); \
          if (~rdy2 && U >= 88) fl2 = __builtin_amdgcn_raw_buffer_load_b128(rl, lane * 16 + 2048, fo, AUX_SYS); FENCE(); } while (0)
#define FLAGS_EVAL() do { rdy0 = __ballot(fl0[0] == call && fl0[2] == call); rdy1 = __ballot(fl1[0] == call && fl1[2] == call); rdy2 = __ballot(fl2[0] == call && fl2[2] == call); } while (0)
        for (;;) {
          asm volatile("s_waitcnt vmcnt(0)" : "+v"(tk) :: "memory"); __builtin_amdgcn_s_waitcnt(0x0F70);
          if (prv && lane == 0) add_ag(wsl + prv_c, 1u);
          u32x4 o[4]; const unsigned ho = hx;
#pragma unroll
          for (int n = 0; n < 4; ++n) o[n] = v[n];
          bool nxt = false; unsigned nxt_d = 0, nxt_c = 0, nxt_m = 0;
          if (more) {
            const unsigned n = (unsigned)uni((int)tk) - tbase;
            if (n < (unsigned)TASKS_IN) {
              nxt = true;
              const int t = n / 192, r = n % 192, u0 = r >> 3, s = r & 7;
              const int u = u0 < 8 ? ((u0 + rel) & 7) : 8 + ((u0 - 8 + (rel * 16) / 7) & 15);
              const int U = t * 24 + u, kt = t * 8 + src;
              while (!(((U < 64 ? rdy0 : U < 128 ? rdy1 : rdy2) >> (U & 63)) & 1)) {
                FLAGS_LOAD(); FLAGS_EVAL();
                if (give_up(wsl, call, t0)) { rdy0 = rdy1 = rdy2 = ~0ull; bad = true; break; }
              }
              const unsigned pl = s < 4 ? (U < 64 ? fl0[1] : U < 128 ? fl1[1] : fl2[1]) : (U < 64 ? fl0[3] : U < 128 ? fl1[3] : fl2[3]);
              nxt_m = ((unsigned)__builtin_amdgcn_readlane((int)pl, U & 63) >> (8 * (s & 3))) & 0xffu;
              const unsigned e = (unsigned)((t * 80 + (u < 8 ? rank * 8 + u : 56 + u)) * 8 + s);
              v[0] = __builtin_amdgcn_raw_buffer_load_b128(rs, lane * 16, OFF_XLE + e * 3072u, AUX_SYS);
              v[1] = __builtin_amdgcn_raw_buffer_load_b128(rs, lane * 16 + 1024, OFF_XLE + e * 3072u, AUX_SYS);
              const bool raw = nxt_m == 127u;
              v[2] = __builtin_amdgcn_raw_buffer_load_b128(rs, lane * 16, raw ? OFF_XH + e * 2048u : OFF_XLE + e * 3072u + 2048u, AUX_SYS);
              v[3] = __builtin_amdgcn_raw_buffer_load_b128(mk_rsrc(wsr, raw ? WS_NEED : 0u), lane * 16 + 1024, OFF_XH + e * 2048u, AUX_SYS);
              hx = __builtin_amdgcn_raw_buffer_load_b32(mk_rsrc(wsr, (nxt_m & 0x80u) ? OFF_XX + e * 64u + 64u : 0u), lane * 4, OFF_XX + e * 64u, AUX_SYS);
              const int grp = (u - 8) * 4 + (s & 3);
              nxt_d = u < 8 ? OFF_STB + (unsigned)(kt * 8 + u) * 32768u + s * 4096u : OFF_STA + (unsigned)((kt * 2 + (s >> 2)) * 64 + grp) * 4096u;
              nxt_c = (u < 8 ? OFF_CNTB + u * CSTR : OFF_CNTA + (grp / 3) * CSTR) + kt * 4;
              if (lane == 0) asm volatile("global_atomic_add %0, %1, %2, off sc0 sc1" : "=v"(tk) : "v"(tp), "v"(1u) : "memory");
              if (!cur) STAMP(1);
            } else more = false;
          }
          FENCE();
          if (cur) {
            unsigned H[8];
            if (cur_m == 127u) {
#pragma unroll
              for (int q = 0; q < 4; ++q) { H[q] = o[2][q]; H[q + 4] = o[3][q]; }
            } else {
              const unsigned R = (cur_m & 0x7fu) * 0x01010101u;
#pragma unroll
              for (int q = 0; q < 4; ++q) {
                H[2 * q] = (R - (o[2][q] & 0x07070707u)) | ((o[2][q] << 4) & 0x80808080u);
                H[2 * q + 1] = (R - ((o[2][q] >> 4) & 0x07070707u)) | (o[2][q] & 0x80808080u);
              }
              if (cur_m & 0x80u) {
                const int ne = uni((int)ho) & 15;
                for (int x = 1; x <= ne; ++x) {
                  const unsigned en = (unsigned)__builtin_amdgcn_readlane((int)ho, x);
                  const unsigned sh = (en >> 16) & 24u, val = (en >> 24) << sh, q0 = (en >> 8) & 7u;
                  const unsigned mk = (unsigned)lane == (en & 63u) ? 0xffu << sh : 0u;
#pragma unroll
                  for (int q = 0; q < 8; ++q) { const unsigned mq = q0 == (unsigned)q ? mk : 0u; H[q] = (H[q] & ~mq) | (val & mq); }
                }
              }
            }
#pragma unroll
            for (int n = 0; n < 4; ++n) {
              u32x4 w;
              const unsigned l0 = o[n >> 1][(n & 1) * 2], l1 = o[n >> 1][(n & 1) * 2 + 1];
              w[0] = __builtin_amdgcn_perm(H[2 * n], l0, 0x05010400u); w[1] = __builtin_amdgcn_perm(H[2 * n], l0, 0x07030602u);
              w[2] = __builtin_amdgcn_perm(H[2 * n + 1], l1, 0x05010400u); w[3] = __builtin_amdgcn_perm(H[2 * n + 1], l1, 0x07030602u);
              __builtin_amdgcn_raw_buffer_store_b128(w, rl, lane * 16 + n * 1024, cur_d, AUX_AG);
            }
          } else if (!nxt) break;
          prv = cur; prv_c = cur_c;
          cur = nxt; cur_d = nxt_d; cur_c = nxt_c; cur_m = nxt_m;
        }
        STAMP(2);
      }
    } else {
      for (int U = 2 * NGEMM + b - NLINKWG; U < NUNIT; U += NCONVWG) conv_unit(P, wsl, lds, U, wid, lane, prev);
      STAMP(1);
    }
    if (wid < 4 && b < 7 * DPL / 4) {
      const unsigned call = (unsigned)uni((int)prev) + 1u;
      const int dw = b * 4 + wid, rel = dw % 7, k = dw / 7;
      const int src = (rank + 1 + rel) & 7;
      const int nsub = (NSUB - k + DPL - 1) / DPL;
      const u64 all = (1ull << nsub) - 1ull;
      char* wsr = P.ws[src];
      __amdgpu_buffer_rsrc_t rs = mk_rsrc(wsr, WS_NEED), rd = mk_rsrc(P.out, OUT_BYTES);
      LAS char* mb = lds + wid * 32768;
      LAS char* sb = mb + 24576;
      const int fsub = lane < nsub ? k + DPL * lane : 0;
      const int ftile = (fsub >> 6) * 8 + ((fsub & 31) >> 2);
      const char* mfp = wsl + OFF_MFLAG + (src * 256 + ftile) * 4;
      const char* vfp = wsl + OFF_VFLAG + (src * 256 + ftile) * 4;
      int cntv = 0;
      unsigned pollv = 0;
      {
        u64 todo = all, ready = 0;
        const u64 t0 = NOW();
        DBG(bool first = true;)
        while (todo) {
          u64 avail = ready & todo;
          if (!avail) {
            ready = __ballot(ld_sys(mfp) == call);
            if (!(ready & todo) && give_up(wsl, call, t0)) { ready = ~0ull; bad = true; }
            continue;
          }
          DBG(if (first) STAMP(3); first = false;)
          int id[4]; u32x3 mv[4];
#pragma unroll
          for (int x = 0; x < 4; ++x) {
            id[x] = avail ? __builtin_ctzll(avail) : -1; avail &= avail - 1;
            mv[x] = __builtin_amdgcn_raw_buffer_load_b96(rs, lane * 12, id[x] >= 0 ? OFF_EXM + (unsigned)(k + DPL * id[x]) * 768u : WS_NEED, AUX_SYS);
          }
          todo &= ~ready | avail;
          pollv = ld_sys(todo ? mfp : vfp);
#pragma unroll
          for (int x = 0; x < 4; ++x) {
            if (id[x] >= 0) {
              *(LAS u32x3a4*)(mb + id[x] * 768 + lane * 12) = mv[x];
              const unsigned sc = scan_incl(__builtin_popcount(mv[x][0]) + __builtin_popcount(mv[x][1]) + __builtin_popcount(mv[x][2]));
              const int tot = __builtin_amdgcn_readlane((int)sc, 63); cntv = lane == id[x] ? tot : cntv;
            }
          }
          if (todo) ready |= __ballot(pollv == call);
        }
      }
      STAMP(4);
      {
        u64 todo = all, ready = 0; int cur = -1; unsigned curcnt = 0;
        u32x3 v[6]; unsigned hv = 0;
        u32x4 q[6]; unsigned qoff = 0; bool pend = false;
        const u64 t0 = NOW();
        const int fr = lane & 15, fq = lane >> 4;
        const unsigned ovoff = (unsigned)fr * GN + fq * 16;
        for (;;) {
          WAIT_V0();
          ready |= __ballot(pollv == call);
          u32x3 o[6]; const unsigned ho = hv;
#pragma unroll
          for (int c = 0; c < 6; ++c) o[c] = v[c];
          const u64 avail = ready & todo;
          int nxt = -1; unsigned nxtcnt = 0;
          if (avail) {
            nxt = __builtin_ctzll(avail); todo &= ~(1ull << nxt);
            nxtcnt = (unsigned)__builtin_amdgcn_readlane(cntv, nxt);
            const unsigned jn = (unsigned)(k + DPL * nxt), so = OFF_EXV + jn * 4608u;
            __amdgpu_buffer_rsrc_t rv = mk_rsrc(wsr, so + ((nxtcnt + 15u) >> 4) * 12u);
            hv = __builtin_amdgcn_raw_buffer_load_b16(mk_rsrc(wsr, OFF_EXH + jn * 64u + 64u), lane * 2, OFF_EXH + jn * 64u, AUX_SYS);
#pragma unroll
            for (int c = 0; c < 6; ++c) v[c] = __builtin_amdgcn_raw_buffer_load_b96(rv, lane * 12 + c * 768, so, AUX_SYS);
          }
          if (todo & ~ready) pollv = ld_sys(vfp);
          FENCE();
          if (pend) {
#pragma unroll
            for (int m = 0; m < 6; ++m) __builtin_amdgcn_raw_buffer_store_b128(q[m], rd, ovoff + m * 16 * GN, qoff, AUX_OUT);
            pend = false;
          }
          FENCE();
          if (cur >= 0) {
            const int j = k + DPL * cur;
            const u32x3 mk = *(const LAS u32x3a4*)(mb + cur * 768 + lane * 12);
            const unsigned nexc = (unsigned)uni((int)ho);
            const bool dense = nexc > 31u;
#pragma unroll
            for (int c = 0; c < 6; ++c) {
              if (c * 1024u < curcnt) {
                u32x4 d;
                d[0] = o[c][0] & 0x3f3f3f3fu; d[1] = o[c][1] & 0x3f3f3f3fu; d[2] = o[c][2] & 0x3f3f3f3fu;
                d[3] = ((o[c][0] >> 6) & 0x03030303u) | ((o[c][1] >> 4) & 0x0c0c0c0cu) | ((o[c][2] >> 2) & 0x30303030u);
                if (dense) {
                  const unsigned h = __builtin_amdgcn_raw_buffer_load_b16(rs, lane * 2 + c * 128, OFF_EXD + (unsigned)j * 768u, AUX_SYS);
                  const unsigned hb = (h & 0x0f0fu) | ((h & 0xf0f0u) << 12);
                  d[0] |= (hb << 6) & 0x40404040u; d[1] |= (hb << 5) & 0x40404040u; d[2] |= (hb << 4) & 0x40404040u; d[3] |= (hb << 3) & 0x40404040u;
                }
                *(LAS u32x4*)(sb + c * 1024 + lane * 16) = d;
              }
            }
            if (!dense && lane >= 1 && (unsigned)lane <= nexc)
              asm volatile("ds_or_b32 %0, %1" :: "v"((unsigned)(size_t)sb + (ho & 0x1ffcu)), "v"(0x40u << (8 * (ho & 3u))) : "memory");
            unsigned off[6];
            (void)piece_offsets(mk, off);
            qoff = (unsigned)((j >> 5) * 96) * GN + src * NLOC + (j & 31) * 64;
#pragma unroll
            for (int m = 0; m < 6; ++m) {
              unsigned a = off[m];
#pragma unroll
              for (int r = 0; r < 4; ++r) {
                const int kk = (m & 1) * 4 + r;
                const unsigned g = (kk >= 2 ? mk[m >> 1] >> (kk - 2) : mk[m >> 1] << (2 - kk)) & 0x04040404u;
                const u32x2 sv = *(const LAS u32x2a4*)(sb + (a & ~3u));
                const unsigned s1 = __builtin_amdgcn_alignbyte(sv[1], sv[0], a);
                a += __builtin_popcount(g);
                q[m][r] = __builtin_amdgcn_perm(0u, s1, g * 0x40404040u + (g ^ 0x04040404u));
              }
            }
            if (bad) q[0][0] = 0x80808080u;
            pend = true;
          } else if (nxt < 0) {
            if (!todo) break;
            if (give_up(wsl, call, t0)) { ready = ~0ull; bad = true; }
          }
          cur = nxt; curcnt = nxtcnt;
        }
      }
      STAMP(6);
    }
  } else {
    const int g = b - GEMM0;
    conv_unit(P, wsl, lds, g, wid, lane, prev);
    STAMP(1);
    conv_unit(P, wsl, lds, NGEMM + g, wid, lane, prev);
    STAMP(6);
    const unsigned call = (unsigned)uni((int)prev) + 1u;
    const int L = (g & 7) * 22 + (g >> 3);
    const int pm = (L % 88) >> 2, pn = (L / 88) * 4 + (L & 3);
    const int wr = wid >> 2, wc = wid & 3;
    volatile LAS int* ctl = (volatile LAS int*)(lds + LDS_CTL);
    __amdgpu_buffer_rsrc_t rl = mk_rsrc(wsl, WS_NEED);
    const unsigned wantA = call * (pm == 21 ? 2u : 6u), wantB = call * 8u;
    f32x4 acc[6][4];
#pragma unroll
    for (int m = 0; m < 6; ++m)
#pragma unroll
      for (int n = 0; n < 4; ++n) acc[m][n] = (f32x4){0.f, 0.f, 0.f, 0.f};
    const unsigned gvoff = lane * 16;
    const bool ldr = wid < 7, real = wid >= 3 || wid == 0 || pm != 21;
    const unsigned gbase = wid < 3 ? OFF_STA + pm * 12288u + wid * 4096u : OFF_STB + pn * 32768u + (wid - 3) * 4096u;
    const unsigned gk = wid < 3 ? 524288u : 262144u, gh = wid < 3 ? 262144u : 16384u;
    const int gdst = wid < 3 ? wid * 4096 : 16384 + (wid - 3) * 4096;
    const int aoff = wr * 6144 + lane * 16, boff = 16384 + wc * 4096 + lane * 16;
    const unsigned pcopy = (unsigned)(size_t)(lds + LDS_CTL + 1536);
#define GL(slot, h, i) do { if (ldr) __builtin_amdgcn_raw_ptr_buffer_load_lds(rl, lds + (slot) * 32768 + gdst, 16, gvoff, so_ + (h) * gh, (i) * 1024, 0); } while (0)
#define POLL() do { if ((ready | issued) != ~0ull) { WAIT_V(6); \
      __builtin_amdgcn_raw_ptr_buffer_load_lds(rl, lds + LDS_CTL + 1536, 4, lane * 4, OFF_CNTA + pm * CSTR, 0, AUX_AG); \
      __builtin_amdgcn_raw_ptr_buffer_load_lds(rl, lds + LDS_CTL + 1792, 4, lane * 4, OFF_CNTB + pn * CSTR, 0, AUX_AG); } } while (0)
#define DECIDE(idle) do { unsigned pa, pb; \
      asm volatile("ds_read_b32 %0, %2\n\tds_read_b32 %1, %2 offset:256\n\ts_waitcnt lgkmcnt(0)" : "=&v"(pa), "=&v"(pb) : "v"(pcopy + lane * 4) : "memory"); \
      ready |= __ballot(pa >= wantA && pb >= wantB); \
      u64 avail = ready & ~issued; \
      if ((idle) && !avail && give_up(wsl, call, t0)) { ready = ~0ull; avail = ~issued; } \
      int kn = -1; \
      if (avail) { kn = __builtin_ctzll(avail); issued |= 1ull << kn; } \
      if (lane == 0) ctl[p & 1] = kn; } while (0)
#define GWAIT() do { if (ldr) WAIT_V(8); } while (0)
#define SETK() do { k2 = uni(k2v); so_ = (k2 >= 0 && real) ? gbase + (unsigned)k2 * gk : WS_NEED; } while (0)
#define RD(NA, NB, rdbase, e) do { if ((e) < 4) NB[(e) & 3] = *(const LAS bf16x8*)((rdbase) + boff + ((e) & 3) * 1024); else NA[((e) + 2) % 6] = *(const LAS bf16x8*)((rdbase) + aoff + (((e) + 2) % 6) * 1024); } while (0)
#define STEP(CA, CB, NA, NB, rdbase, wslot, h) do { \
      _Pragma("unroll") for (int m = 0; m < 6; ++m) { \
        if (m == 0 && (h) == 0) k2v = ctl[(p & 1) ^ 1]; \
        if (m < 5) { RD(NA, NB, rdbase, 2 * m); RD(NA, NB, rdbase, 2 * m + 1); } \
        _Pragma("unroll") for (int n = 0; n < 4; ++n) acc[m][n] = __builtin_amdgcn_mfma_f32_16x16x32_bf16(CB[n], CA[m], acc[m][n], 0, 0, 0); \
        SCHED(); \
        if (m == 2 && (h) == 1 && !ldr) { DECIDE(false); POLL(); } \
        if (m == 1 && (h) == 0) SETK(); \
        if (m == 1) GL(wslot, h, 0); if (m == 2) GL(wslot, h, 1); if (m == 3) GL(wslot, h, 2); if (m == 4) GL(wslot, h, 3); \
        SCHED(); \
      } } while (0)
    bf16x8 xa[6], xb[4], ya[6], yb[4];
#pragma unroll
    for (int m = 0; m < 6; ++m) xa[m] = (bf16x8)(__bf16)0.f;
#pragma unroll
    for (int n = 0; n < 4; ++n) xb[n] = (bf16x8)(__bf16)0.f;
    u64 issued = 0, ready = 0;
    int k0 = -1, k1 = -1, k2 = -1, k2v = -1, ndone = 0;
    if (tid == 0) ctl[1] = -1;
    if (tid < 128) ctl[384 + tid] = 0;
    const u64 t0 = NOW();
    BAR();
    for (int p = 0;; ++p) {
      const int s0 = (p & 1) * 2;
      unsigned so_;
      if (k0 >= 0) {
        STEP(xa, xb, ya, yb, lds + (s0 + 1) * 32768, s0, 0);
        GWAIT(); BAR();
        STEP(ya, yb, xa, xb, lds + (s0 ^ 2) * 32768, s0 + 1, 1);
        ++ndone;
      } else {
        k2v = ctl[(p & 1) ^ 1];
        SETK();
        if (!ldr) POLL();
        GL(s0, 0, 0); GL(s0, 0, 1); GL(s0, 0, 2); GL(s0, 0, 3);
        GWAIT(); BAR();
#pragma unroll
        for (int e = 0; e < 10; ++e) RD(xa, xb, lds + (s0 ^ 2) * 32768, e);
        GL(s0 + 1, 1, 0); GL(s0 + 1, 1, 1); GL(s0 + 1, 1, 2); GL(s0 + 1, 1, 3);
        if (!ldr) {
          const bool idle = k1 < 0 && k2 < 0;
          if (idle) WAIT_V0();
          DECIDE(idle);
        }
        FENCE(); __builtin_amdgcn_s_waitcnt(0xC07F);
      }
      DBG(if (ndone == 1 && k0 >= 0) STAMP(7); if (ndone == 32 && k0 >= 0) STAMP(8); if (ndone == 56 && k0 >= 0) STAMP(9);)
      if (ndone == 64) { STAMPV(10, (u64)p * 100); break; }
      GWAIT();
      BAR();
      k0 = k1; k1 = k2;
    }
    STAMP(2);
    if (pm == 21) {
#pragma unroll
      for (int m = 0; m < 6; ++m)
        if (wr == 1 || m >= 4) {
#pragma unroll
          for (int n = 0; n < 4; ++n) acc[m][n] = (f32x4){-1.f, -1.f, -1.f, -1.f};
        }
    }
    const int tile = pm * 8 + pn, sub = (pm * 2 + wr) * 32 + pn * 4 + wc;
    u32x3 mk;
#pragma unroll
    for (int D = 0; D < 3; ++D) {
      unsigned mm = 0;
#pragma unroll
      for (int n = 3; n >= 0; --n)
#pragma unroll
        for (int kk = 7; kk >= 0; --kk) mm = __builtin_amdgcn_alignbit(mm, __float_as_uint(acc[2 * D + (kk >> 2)][n][kk & 3]), 31);
      mk[D] = ~mm;
    }
    __builtin_amdgcn_raw_buffer_store_b96(mk, rl, lane * 12, OFF_EXM + (unsigned)sub * 768u, AUX_SYS);
    float mx = 0.f;
#pragma unroll
    for (int m = 0; m < 6; ++m)
#pragma unroll
      for (int n = 0; n < 4; ++n) mx = fmaxf(fmaxf(mx, fmaxf(acc[m][n][0], acc[m][n][1])), fmaxf(acc[m][n][2], acc[m][n][3]));
#pragma unroll
    for (int o = 32; o >= 1; o >>= 1) mx = fmaxf(mx, __shfl_xor(mx, o));
    volatile LAS float* red = (volatile LAS float*)(lds + LDS_CTL + 64);
    if (lane == 0) red[wid] = mx;
    WAIT_V0(); BAR();
    if (wid == 1 && lane < NR && lane != rank) st_sys(P.ws[lane] + OFF_MFLAG + (rank * 256 + tile) * 4, call);
    STAMP(3);
    if (wid == 0) {
      float v = red[lane & 7];
#pragma unroll
      for (int o = 4; o >= 1; o >>= 1) v = fmaxf(v, __shfl_xor(v, o));
      unsigned last = 0;
      if (lane == 0) {
        (void)__hip_atomic_fetch_max((GAS u64*)(wsl + OFF_AMAXL), ((u64)call << 32) | __float_as_uint(v), __ATOMIC_RELAXED, __HIP_MEMORY_SCOPE_AGENT);
        WAIT_V0();
        last = __hip_atomic_fetch_add((GAS unsigned*)(wsl + OFF_AMAXT), 1u, __ATOMIC_RELAXED, __HIP_MEMORY_SCOPE_AGENT) + 1u == call * (unsigned)NGEMM;
      }
      if (uni((int)last)) {
        const u64 gm = __hip_atomic_load((GAS u64*)(wsl + OFF_AMAXL), __ATOMIC_RELAXED, __HIP_MEMORY_SCOPE_AGENT);
        if (lane < NR) st_sys64(P.ws[lane] + OFF_AMAXG + rank * 128, gm);
      }
      const u64 t1 = NOW(); u64 gv;
      for (;;) {
        gv = ld_sys64(wsl + OFF_AMAXG + (lane & 7) * 128);
        if (__all((unsigned)(gv >> 32) == call)) break;
        if (give_up(wsl, call, t1)) break;
      }
      float am = __uint_as_float((unsigned)gv);
#pragma unroll
      for (int o = 4; o >= 1; o >>= 1) am = fmaxf(am, __shfl_xor(am, o));
      const unsigned tm = ld_ag(wsl + OFF_TMO);
      if (lane == 0) { red[8] = am; red[9] = (tm == call) ? 1.f : 0.f; }
    }
    unsigned off[6];
    const unsigned cnt = piece_offsets(mk, off);
    BAR();
    STAMP(4);
    const float amax = red[8]; const bool bad = red[9] != 0.f;
    const float inv = amax > 0.f ? 127.f / amax : 0.f;
    u32x4 q[6];
#pragma unroll
    for (int m = 0; m < 6; ++m) {
      unsigned a = wid * 8192 + off[m];
#pragma unroll
      for (int r = 0; r < 4; ++r) {
        unsigned d = 0;
#pragma unroll
        for (int n = 0; n < 4; ++n) d = __builtin_amdgcn_cvt_pk_u8_f32(__builtin_rintf(acc[m][n][r] * inv), n, d);
        q[m][r] = d;
        const unsigned d8 = d >> 8;
#pragma unroll
        for (int n = 0; n < 4; ++n) {
          const int sg = (int)__float_as_uint(acc[m][n][r]) >> 31;
          lds[a | ((unsigned)sg & 0x10000u)] = (char)(((n & 1) ? d8 : d) >> ((n & 2) * 8));
          a += 1 + sg;
        }
      }
    }
    {
      LAS unsigned short* hd = (LAS unsigned short*)(lds + LDS_CTL + 256 + wid * 128);
      const unsigned so = OFF_EXV + (unsigned)sub * 4608u;
      unsigned nexc = 0;
#pragma unroll 1
      for (unsigned c = 0; c * 1024u < cnt; ++c) {
        const u32x4 d = *(const LAS u32x4*)(lds + wid * 8192 + c * 1024 + lane * 16);
        u32x3 o;
        o[0] = (d[0] & 0x3f3f3f3fu) | ((d[3] << 6) & 0xc0c0c0c0u);
        o[1] = (d[1] & 0x3f3f3f3fu) | ((d[3] << 4) & 0xc0c0c0c0u);
        o[2] = (d[2] & 0x3f3f3f3fu) | ((d[3] << 2) & 0xc0c0c0c0u);
        __builtin_amdgcn_raw_buffer_store_b96(o, rl, lane * 12 + c * 768, so, AUX_SYS);
        unsigned hb = ((d[0] >> 6) & 0x01010101u) | ((d[1] >> 5) & 0x02020202u) | ((d[2] >> 4) & 0x04040404u) | ((d[3] >> 3) & 0x08080808u);
        __builtin_amdgcn_raw_buffer_store_b16((unsigned short)((hb & 0x0f0fu) | ((hb >> 12) & 0xf0f0u)), rl, lane * 2 + c * 128, OFF_EXD + (unsigned)sub * 768u, AUX_SYS);
        const unsigned pos0 = c * 1024 + lane * 16;
        if (pos0 >= cnt) hb = 0;
        for (;;) {
          const u64 bal = __ballot(hb != 0);
          if (!bal || nexc > 31u) break;
          const unsigned sl = nexc + __builtin_amdgcn_mbcnt_hi((unsigned)(bal >> 32), __builtin_amdgcn_mbcnt_lo((unsigned)bal, 0u));
          if (hb != 0 && sl < 31u) { const unsigned bi = __builtin_ctz(hb); hd[1 + sl] = (unsigned short)(pos0 + 4 * (bi & 3) + (bi >> 3)); }
          hb &= hb - 1;
          nexc += __builtin_popcountll(bal);
        }
      }
      if (lane == 0) hd[0] = (unsigned short)(nexc > 31u ? 0xffffu : nexc);
      if (lane < 16) __builtin_amdgcn_raw_buffer_store_b32(((LAS unsigned*)hd)[lane], rl, lane * 4, OFF_EXH + (unsigned)sub * 64u, AUX_SYS);
    }
    WAIT_V0(); BAR();
    if (tid < NR && tid != rank) st_sys(P.ws[tid] + OFF_VFLAG + (rank * 256 + tile) * 4, call);
    STAMP(5);
    {
      __amdgpu_buffer_rsrc_t rd = mk_rsrc(P.out, OUT_BYTES);
      const unsigned ovoff = (unsigned)(wr * 96 + (lane & 15)) * GN + wc * 64 + (lane >> 4) * 16;
      const unsigned osoff = (unsigned)(pm * 192) * GN + rank * NLOC + pn * 256;
#pragma unroll
      for (int m = 0; m < 6; ++m) {
        if (bad) q[m][0] = 0x80808080u;
        __builtin_amdgcn_raw_buffer_store_b128(q[m], rd, ovoff + m * 16 * GN, osoff, AUX_OUT);
      }
    }
  }
  __syncthreads();
  if (tid == 0) st_ag(wsl + OFF_CALL + b * 64, prev + 1u);
}
}

extern "C" void kernel_launch(int rank, int nranks, ncclComm_t comm, hipStream_t stream,
                              void* const* d_in, const int* in_sizes, int n_in,
                              void* d_out, int out_size, const int* in_dtypes, int out_dtype,
                              void* const* d_out_all, void* const* d_ws_all, size_t ws_bytes) {
  if (nranks != NR || (unsigned)rank >= (unsigned)NR || ws_bytes < WS_NEED || out_size != GM * GN) { fprintf(stderr, "unexpected configuration\n"); return; }
  Params P;
  P.x = (const float*)d_in[0]; P.w = (const float*)d_in[1]; P.out = (char*)d_out;
  for (int p = 0; p < NR; ++p) P.ws[p] = (char*)d_ws_all[p];
  P.rank = rank;
  hipLaunchKernelGGL(fused_kernel, dim3(NWG), dim3(512), 0, stream, P);
}
```

```cpp
#include <hip/hip_runtime.h>
#include <rccl/rccl.h>
#include <cstdio>

#define LAS __attribute__((address_space(3)))
#define GAS __attribute__((address_space(1)))
typedef unsigned u32x4 __attribute__((ext_vector_type(4)));
typedef unsigned u32x3 __attribute__((ext_vector_type(3)));
typedef unsigned u32x2 __attribute__((ext_vector_type(2)));
typedef u32x2 u32x2a4 __attribute__((aligned(4)));
typedef u32x3 u32x3a4 __attribute__((aligned(4)));
typedef float f32x4 __attribute__((ext_vector_type(4)));
typedef float f32x2 __attribute__((ext_vector_type(2)));
typedef __bf16 bf16x2 __attribute__((ext_vector_type(2)));
typedef __bf16 bf16x8 __attribute__((ext_vector_type(8)));
typedef unsigned long long u64;

namespace {
constexpr int NR = 8;
constexpr int GM = 4096, GN = 16384, KSH = 512, NLOC = 2048;
constexpr int NWG = 256, NLINKWG = 56, NCONVWG = 24, GEMM0 = 80, NGEMM = 176;
constexpr int NSUB = 1376;
#define PWV 4
constexpr int WPL = PWV * 8;
constexpr int DPL = 44;
constexpr int TASKS_IN = 1536;
constexpr int NUNIT = 640;
constexpr unsigned MiB = 1u << 20;
constexpr unsigned OFF_CALL = 0;
constexpr unsigned OFF_TMO = 16384;
constexpr unsigned OFF_AMAXL = 16384 + 256;
constexpr unsigned OFF_AMAXT = 16384 + 512;
constexpr unsigned OFF_AMAXG = 16384 + 1024;
constexpr unsigned CSTR = 4352;
constexpr unsigned OFF_CNTA = 131072;
constexpr unsigned OFF_CNTB = OFF_CNTA + 22 * CSTR;
constexpr unsigned OFF_TICK = OFF_CNTB + 8 * CSTR;
constexpr unsigned OFF_MFLAG = 49152;
constexpr unsigned OFF_VFLAG = 49152 + 8192;
constexpr unsigned OFF_UFLAG = 65536;
constexpr unsigned OFF_DBG = 524288;
constexpr unsigned OFF_STA = 1 * MiB;
constexpr unsigned OFF_STB = 33 * MiB;
constexpr unsigned OFF_XLE = 49 * MiB;
constexpr unsigned OFF_XH = 64 * MiB;
constexpr unsigned OFF_XX = 74 * MiB;
constexpr unsigned OFF_EXM = 75 * MiB;
constexpr unsigned OFF_EXV = 77 * MiB;
constexpr unsigned OFF_EXD = 84 * MiB;
constexpr unsigned OFF_EXH = 86 * MiB;
constexpr unsigned WS_NEED = 87 * MiB;
constexpr unsigned OUT_BYTES = (unsigned)GM * GN;
constexpr u64 DEADLINE = 50000000ull;
constexpr int SLOT = 28672, NS = 5;
constexpr int LDS_CTL = NS * SLOT;
constexpr int LDS_BYTES = LDS_CTL + 2048;
constexpr int AUX_SYS = 17, AUX_AG = 16;
#define POLL_COST 4
#define POLL_SLOW 16
#define FAST_AT 104
#define IDLE_SLEEP 40
#define PRIO_MM 0
#define PRIO_OFF 1
#define START_SLEEP 50
#define AUX_OUT 18

struct Params {
  const float* x; const float* w;
  char* out; char* ws[NR];
  int rank;
};

__device__ __forceinline__ unsigned ld_sys(const char* p) { return __hip_atomic_load((const GAS unsigned*)p, __ATOMIC_RELAXED, __HIP_MEMORY_SCOPE_SYSTEM); }
__device__ __forceinline__ u64 ld_sys64(const char* p) { return __hip_atomic_load((const GAS u64*)p, __ATOMIC_RELAXED, __HIP_MEMORY_SCOPE_SYSTEM); }
__device__ __forceinline__ unsigned ld_ag(const char* p) { return __hip_atomic_load((const GAS unsigned*)p, __ATOMIC_RELAXED, __HIP_MEMORY_SCOPE_AGENT); }
__device__ __forceinline__ void st_sys(char* p, unsigned v) { __hip_atomic_store((GAS unsigned*)p, v, __ATOMIC_RELAXED, __HIP_MEMORY_SCOPE_SYSTEM); }
__device__ __forceinline__ void st_sys64(char* p, u64 v) { __hip_atomic_store((GAS u64*)p, v, __ATOMIC_RELAXED, __HIP_MEMORY_SCOPE_SYSTEM); }
__device__ __forceinline__ void st_ag(char* p, unsigned v) { __hip_atomic_store((GAS unsigned*)p, v, __ATOMIC_RELAXED, __HIP_MEMORY_SCOPE_AGENT); }
__device__ __forceinline__ void add_ag(char* p, unsigned v) { (void)__hip_atomic_fetch_add((GAS unsigned*)p, v, __ATOMIC_RELAXED, __HIP_MEMORY_SCOPE_AGENT); }
__device__ __forceinline__ int uni(int v) { return __builtin_amdgcn_readfirstlane(v); }
__device__ __forceinline__ __amdgpu_buffer_rsrc_t mk_rsrc(const void* p, unsigned bytes) { return __builtin_amdgcn_make_buffer_rsrc((void*)p, 0, (int)bytes, 0x00020000); }
__device__ __forceinline__ unsigned pk_bf16(float a, float b) { f32x2 v = {a, b}; return __builtin_bit_cast(unsigned, __builtin_convertvector(v, bf16x2)); }
#define WAIT_V(n) asm volatile("s_waitcnt vmcnt(" #n ")" ::: "memory")
#define WAIT_V0() do { WAIT_V(0); __builtin_amdgcn_s_waitcnt(0x0F70); } while (0)
#define FENCE() asm volatile("" ::: "memory")
#define BAR() do { asm volatile("" ::: "memory"); __builtin_amdgcn_s_waitcnt(0xC07F); __builtin_amdgcn_s_barrier(); asm volatile("" ::: "memory"); } while (0)
#define SCHED() __builtin_amdgcn_sched_barrier(0)
#define NOW() __builtin_amdgcn_s_memrealtime()
#define ST128(d, r, vo, so, aux) do { const u32x4 d_ = (d); __builtin_amdgcn_raw_buffer_store_b128(d_, r, vo, so, aux); asm volatile("s_nop 1" :: "v"(d_) : "memory"); } while (0)
#define ST96(d, r, vo, so, aux) do { const u32x3 d_ = (d); __builtin_amdgcn_raw_buffer_store_b96(d_, r, vo, so, aux); asm volatile("s_nop 1" :: "v"(d_) : "memory"); } while (0)

__device__ __forceinline__ bool give_up(char* wsl, unsigned call, u64 t0) {
  const u64 el = NOW() - t0;
  if (el > 100000ull && (el > DEADLINE || ld_ag(wsl + OFF_TMO) == call)) { st_ag(wsl + OFF_TMO, call); return true; }
  __builtin_amdgcn_s_sleep(2);
  return false;
}

#define STAMP(i) do {} while (0)
#define STAMPV(i, v) do {} while (0)
#define DBG(x)
#define DBG2(x)

__device__ __forceinline__ unsigned scan_incl(unsigned v) {
  v += __builtin_amdgcn_update_dpp(0, (int)v, 0x111, 0xf, 0xf, false);
  v += __builtin_amdgcn_update_dpp(0, (int)v, 0x112, 0xf, 0xf, false);
  v += __builtin_amdgcn_update_dpp(0, (int)v, 0x114, 0xf, 0xf, false);
  v += __builtin_amdgcn_update_dpp(0, (int)v, 0x118, 0xf, 0xf, false);
  v += __builtin_amdgcn_update_dpp(0, (int)v, 0x142, 0xa, 0xf, false);
  v += __builtin_amdgcn_update_dpp(0, (int)v, 0x143, 0xc, 0xf, false);
  return v;
}
__device__ __forceinline__ unsigned piece_offsets(const u32x3 mk, unsigned (&off)[6]) {
  unsigned base = 0;
#pragma unroll
  for (int D = 0; D < 3; ++D) {
    const unsigned c = __builtin_popcount(mk[D] & 0x0f0f0f0fu) | (__builtin_popcount(mk[D] & 0xf0f0f0f0u) << 16);
    const unsigned s = scan_incl(c), ex = s - c, tot = (unsigned)__builtin_amdgcn_readlane((int)s, 63);
    off[2 * D] = base + (ex & 0xffffu); base += tot & 0xffffu;
    off[2 * D + 1] = base + (ex >> 16); base += tot >> 16;
  }
  return base;
}

__device__ __forceinline__ void conv_unit(const Params& P, char* wsl, LAS char* lds, int U, int wid, int lane, unsigned prev) {
  const int rank = P.rank;
  const int t = U / 80, r = U % 80, s = wid;
  const int kt = t * 8 + rank;
  const int c = lane & 15, kg = lane >> 4;
  const bool isB = r < 64;
  const int dest = (rank + 1 + (r & 7)) & 7, j = r >> 3, i = r - 64;
  __amdgpu_buffer_rsrc_t rl = mk_rsrc(wsl, WS_NEED);
  unsigned f[16];
  if (isB) {
    const GAS float* src = (const GAS float*)P.w + (size_t)(t * 64 + (s >> 2) * 32 + kg * 8) * GN + dest * NLOC + j * 256 + (s & 3) * 64 + c * 4;
    f32x4 v[8];
#pragma unroll
    for (int q = 0; q < 8; ++q) v[q] = __builtin_nontemporal_load((const GAS f32x4*)(src + (size_t)q * GN));
    SCHED();
#pragma unroll
    for (int n = 0; n < 4; ++n)
#pragma unroll
      for (int d = 0; d < 4; ++d) f[n * 4 + d] = pk_bf16(v[2 * d][n], v[2 * d + 1][n]);
  } else {
    const GAS float* src = (const GAS float*)P.x + (size_t)(i * 256 + (s & 3) * 64 + c) * KSH + t * 64 + (s >> 2) * 32 + kg * 8;
    f32x4 v[8];
#pragma unroll
    for (int q = 0; q < 8; ++q) v[q] = *(const GAS f32x4*)(src + (q >> 1) * 16 * KSH + (q & 1) * 4);
    SCHED();
#pragma unroll
    for (int q = 0; q < 8; ++q) { f[2 * q] = pk_bf16(v[q][0], v[q][1]); f[2 * q + 1] = pk_bf16(v[q][2], v[q][3]); }
  }
  const bool local = !isB || dest == rank, remote = !isB || dest != rank;
  if (remote) {
    const unsigned e = (unsigned)((t * 80 + (isB ? dest * 8 + j : r)) * 8 + s);
    u32x4 L0, L1, E; unsigned H[8];
    unsigned mxv = 0;
#pragma unroll
    for (int q = 0; q < 8; ++q) {
      const unsigned lo = __builtin_amdgcn_perm(f[2 * q + 1], f[2 * q], 0x06040200u);
      H[q] = __builtin_amdgcn_perm(f[2 * q + 1], f[2 * q], 0x07050301u);
      if (q < 4) L0[q] = lo; else L1[q - 4] = lo;
      const unsigned a = f[2 * q] & 0x7fff7fffu, bq = f[2 * q + 1] & 0x7fff7fffu;
      mxv = max(mxv, max(max(a, a << 16), max(bq, bq << 16)));
    }
    ST128(L0, rl, lane * 16, OFF_XLE + e * 3072u, AUX_SYS);
    ST128(L1, rl, lane * 16 + 1024, OFF_XLE + e * 3072u, AUX_SYS);
#pragma unroll
    for (int o = 32; o >= 1; o >>= 1) mxv = max(mxv, (unsigned)__shfl_xor((int)mxv, o));
    unsigned ref = (unsigned)uni((int)(mxv >> 24));
    const unsigned R = ref * 0x01010101u;
    unsigned any = 0;
#pragma unroll
    for (int q = 0; q < 4; ++q) {
      const unsigned d0 = R - (H[2 * q] & 0x7f7f7f7fu), d1 = R - (H[2 * q + 1] & 0x7f7f7f7fu);
      any |= d0 | d1;
      E[q] = (d0 & 0x07070707u) | ((H[2 * q] >> 4) & 0x08080808u) | ((d1 & 0x07070707u) << 4) | (H[2 * q + 1] & 0x80808080u);
    }
    unsigned nexc = 0;
    if (__ballot((any & 0x78787878u) != 0u)) {
      LAS unsigned* hd = (LAS unsigned*)(lds + LDS_CTL + 256 + wid * 128);
#pragma unroll
      for (int q = 0; q < 8; ++q) {
        const unsigned d = R - (H[q] & 0x7f7f7f7fu);
        unsigned ov = (((d & 0x78787878u) + 0x78787878u) & 0x80808080u);
        for (;;) {
          const u64 bal = __ballot(ov != 0u);
          if (!bal || nexc > 15u) break;
          const unsigned sl = nexc + __builtin_amdgcn_mbcnt_hi((unsigned)(bal >> 32), __builtin_amdgcn_mbcnt_lo((unsigned)bal, 0u));
          if (ov != 0u && sl < 15u) { const unsigned sh = __builtin_ctz(ov) & 24u; hd[1 + sl] = (unsigned)lane | (q << 8) | (sh << 16) | (((H[q] >> sh) & 0xffu) << 24); }
          ov &= ov - 1;
          nexc += __builtin_popcountll(bal);
        }
      }
      if (nexc <= 15u) {
        if (lane == 0) hd[0] = nexc;
        if (lane < 16) __builtin_amdgcn_raw_buffer_store_b32(hd[lane], rl, lane * 4, OFF_XX + e * 64u, AUX_SYS);
      }
    }
    if (nexc > 15u || ref == 127u) {
      ref = 127u; nexc = 0;
      const u32x4 H0 = {H[0], H[1], H[2], H[3]}, H1 = {H[4], H[5], H[6], H[7]};
      ST128(H0, rl, lane * 16, OFF_XH + e * 2048u, AUX_SYS);
      ST128(H1, rl, lane * 16 + 1024, OFF_XH + e * 2048u, AUX_SYS);
    } else {
      ST128(E, rl, lane * 16 + 2048, OFF_XLE + e * 3072u, AUX_SYS);
    }
    if (lane == 0) *(volatile LAS unsigned char*)(lds + LDS_CTL + 128 + wid) = (unsigned char)(ref | (nexc ? 0x80u : 0u));
  }
  if (local) {
    const unsigned base = isB ? OFF_STB + (unsigned)(kt * 8 + j) * 32768u + s * 4096u : OFF_STA + (unsigned)((kt * 2 + (s >> 2)) * 64 + i * 4 + (s & 3)) * 4096u;
#pragma unroll
    for (int n = 0; n < 4; ++n) { const u32x4 o = {f[4 * n], f[4 * n + 1], f[4 * n + 2], f[4 * n + 3]}; ST128(o, rl, lane * 16 + n * 1024, base, AUX_AG); }
  }
  WAIT_V0();
  if (local && lane == 0) add_ag(wsl + (isB ? OFF_CNTB + j * CSTR : OFF_CNTA + ((i * 4 + (s & 3)) / 3) * CSTR) + kt * 4, 1u);
  BAR();
  if (wid == 0) {
    if (remote && lane < 2 * NR) {
      const int pd = lane >> 1, hf = lane & 1;
      const unsigned pay = *(volatile LAS unsigned*)(lds + LDS_CTL + 128 + hf * 4);
      if (pd != rank && (!isB || pd == dest))
        st_sys64(P.ws[pd] + OFF_UFLAG + rank * CSTR + (t * 24 + (isB ? j : 8 + i)) * 16 + hf * 8, ((u64)pay << 32) | (prev + 1u));
    }
  }
  BAR();
}

__global__ __launch_bounds__(512, 2) void fused_kernel(Params P) {
  __shared__ __attribute__((aligned(1024))) char smem[LDS_BYTES];
  const int tid = threadIdx.x, lane = tid & 63, wid = uni(tid >> 6);
  const int b = blockIdx.x, rank = P.rank;
  char* wsl = P.ws[rank];
  LAS char* lds = (LAS char*)smem;
  STAMP(0);
  DBG(const u64 tstart = NOW();)
  const unsigned prev = ld_ag(wsl + OFF_CALL + b * 64);

  if (b < GEMM0) {
    bool bad = false;
    if (b < NLINKWG) {
      if (wid < PWV) {
        const unsigned call = (unsigned)uni((int)prev) + 1u;
        const int gw = b * PWV + wid, rel = gw % 7, k = gw / 7;
        const int src = (rank + 1 + rel) & 7;
        __amdgpu_buffer_rsrc_t rs = mk_rsrc(P.ws[src], WS_NEED), rl = mk_rsrc(wsl, WS_NEED);
        const unsigned tbase = (call - 1u) * (unsigned)TASKS_IN - (unsigned)WPL;
        char* tp = wsl + OFF_TICK + rel * CSTR;
        const unsigned fo = OFF_UFLAG + src * CSTR;
        u64 rdy0 = 0, rdy1 = 0, rdy2 = 0;
        u32x4 fl0 = {0, 0, 0, 0}, fl1 = fl0, fl2 = fl0;
        u32x4 v[4] = {fl0, fl0, fl0, fl0}; unsigned tk = tbase + (unsigned)k, hx = 0;
        char* wsr = P.ws[src];
        unsigned cur_d = 0, cur_c = 0, cur_m = 0, prv_c = 0; bool cur = false, prv = false, more = true;
        const u64 t0 = NOW();
#define FLAGS_LOAD() do { \
          if (~rdy0) fl0 = __builtin_amdgcn_raw_buffer_load_b128(rl, lane * 16, fo, AUX_SYS); \
          if (~rdy1 && U >= 24) fl1 = __builtin_amdgcn_raw_buffer_load_b128(rl, lane * 16 + 1024, fo, AUX_SYS); \
          if (~rdy2 && U >= 88) fl2 = __builtin_amdgcn_raw_buffer_load_b128(rl, lane * 16 + 2048, fo, AUX_SYS); FENCE(); } while (0)
#define FLAGS_EVAL() do { rdy0 = __ballot(fl0[0] == call && fl0[2] == call); rdy1 = __ballot(fl1[0] == call && fl1[2] == call); rdy2 = __ballot(fl2[0] == call && fl2[2] == call); } while (0)
        for (;;) {
          if (prv && lane == 0) add_ag(wsl + prv_c, 1u);
          u32x4 o[4]; const unsigned ho = hx;
#pragma unroll
          for (int n = 0; n < 4; ++n) o[n] = v[n];
          bool nxt = false; unsigned nxt_d = 0, nxt_c = 0, nxt_m = 0;
          if (more) {
            const unsigned n = (unsigned)uni((int)tk) - tbase;
            if (n < (unsigned)TASKS_IN) {
              nxt = true;
              const int t = n / 192, r = n % 192, u0 = r >> 3, s = r & 7;
              const int u = u0 < 8 ? ((u0 + rel) & 7) : 8 + ((u0 - 8 + (rel * 16) / 7) & 15);
              const int U = t * 24 + u, kt = t * 8 + src;
              while (!(((U < 64 ? rdy0 : U < 128 ? rdy1 : rdy2) >> (U & 63)) & 1)) {
                FLAGS_LOAD(); FLAGS_EVAL();
                if (give_up(wsl, call, t0)) { rdy0 = rdy1 = rdy2 = ~0ull; bad = true; break; }
              }
              const unsigned pl = s < 4 ? (U < 64 ? fl0[1] : U < 128 ? fl1[1] : fl2[1]) : (U < 64 ? fl0[3] : U < 128 ? fl1[3] : fl2[3]);
              nxt_m = ((unsigned)__builtin_amdgcn_readlane((int)pl, U & 63) >> (8 * (s & 3))) & 0xffu;
              const unsigned e = (unsigned)((t * 80 + (u < 8 ? rank * 8 + u : 56 + u)) * 8 + s);
              v[0] = __builtin_amdgcn_raw_buffer_load_b128(rs, lane * 16, OFF_XLE + e * 3072u, AUX_SYS);
              v[1] = __builtin_amdgcn_raw_buffer_load_b128(rs, lane * 16 + 1024, OFF_XLE + e * 3072u, AUX_SYS);
              const bool raw = nxt_m == 127u;
              v[2] = __builtin_amdgcn_raw_buffer_load_b128(rs, lane * 16, raw ? OFF_XH + e * 2048u : OFF_XLE + e * 3072u + 2048u, AUX_SYS);
              v[3] = __builtin_amdgcn_raw_buffer_load_b128(mk_rsrc(wsr, raw ? WS_NEED : 0u), lane * 16 + 1024, OFF_XH + e * 2048u, AUX_SYS);
              hx = __builtin_amdgcn_raw_buffer_load_b32(mk_rsrc(wsr, (nxt_m & 0x80u) ? OFF_XX + e * 64u + 64u : 0u), lane * 4, OFF_XX + e * 64u, AUX_SYS);
              const int grp = (u - 8) * 4 + (s & 3);
              nxt_d = u < 8 ? OFF_STB + (unsigned)(kt * 8 + u) * 32768u + s * 4096u : OFF_STA + (unsigned)((kt * 2 + (s >> 2)) * 64 + grp) * 4096u;
              nxt_c = (u < 8 ? OFF_CNTB + u * CSTR : OFF_CNTA + (grp / 3) * CSTR) + kt * 4;
              if (lane == 0) asm volatile("global_atomic_add %0, %1, %2, off sc0 sc1" : "=v"(tk) : "v"(tp), "v"(1u) : "memory");
              if (!cur) STAMP(1);
            } else more = false;
          }
          FENCE();
          if (cur) {
            unsigned H[8];
            if (cur_m == 127u) {
#pragma unroll
              for (int q = 0; q < 4; ++q) { H[q] = o[2][q]; H[q + 4] = o[3][q]; }
            } else {
              const unsigned R = (cur_m & 0x7fu) * 0x01010101u;
#pragma unroll
              for (int q = 0; q < 4; ++q) {
                H[2 * q] = (R - (o[2][q] & 0x07070707u)) | ((o[2][q] << 4) & 0x80808080u);
                H[2 * q + 1] = (R - ((o[2][q] >> 4) & 0x07070707u)) | (o[2][q] & 0x80808080u);
              }
              if (cur_m & 0x80u) {
                const int ne = uni((int)ho) & 15;
                for (int x = 1; x <= ne; ++x) {
                  const unsigned en = (unsigned)__builtin_amdgcn_readlane((int)ho, x);
                  const unsigned sh = (en >> 16) & 24u, val = (en >> 24) << sh, q0 = (en >> 8) & 7u;
                  const unsigned mk = (unsigned)lane == (en & 63u) ? 0xffu << sh : 0u;
#pragma unroll
                  for (int q = 0; q < 8; ++q) { const unsigned mq = q0 == (unsigned)q ? mk : 0u; H[q] = (H[q] & ~mq) | (val & mq); }
                }
              }
            }
#pragma unroll
            for (int n = 0; n < 4; ++n) {
              u32x4 w;
              const unsigned l0 = o[n >> 1][(n & 1) * 2], l1 = o[n >> 1][(n & 1) * 2 + 1];
              w[0] = __builtin_amdgcn_perm(H[2 * n], l0, 0x05010400u); w[1] = __builtin_amdgcn_perm(H[2 * n], l0, 0x07030602u);
              w[2] = __builtin_amdgcn_perm(H[2 * n + 1], l1, 0x05010400u); w[3] = __builtin_amdgcn_perm(H[2 * n + 1], l1, 0x07030602u);
              ST128(w, rl, lane * 16 + n * 1024, cur_d, AUX_AG);
            }
          } else if (!nxt) break;
          prv = cur; prv_c = cur_c;
          cur = nxt; cur_d = nxt_d; cur_c = nxt_c; cur_m = nxt_m;
          WAIT_V0();
          asm volatile("" : "+v"(tk), "+v"(hx), "+v"(v[0]), "+v"(v[1]), "+v"(v[2]), "+v"(v[3]));
        }
        STAMP(2);
      }
    } else {
      if (START_SLEEP) __builtin_amdgcn_s_sleep(START_SLEEP);
      for (int U = 2 * NGEMM + b - NLINKWG; U < NUNIT; U += NCONVWG) conv_unit(P, wsl, lds, U, wid, lane, prev);
      STAMP(1);
    }
    if (wid < 4 && b < 7 * DPL / 4) {
      const unsigned call = (unsigned)uni((int)prev) + 1u;
      const int dw = b * 4 + wid, rel = dw % 7, k = dw / 7;
      const int src = (rank + 1 + rel) & 7;
      const int nsub = (NSUB - k + DPL - 1) / DPL;
      const u64 all = (1ull << nsub) - 1ull;
      char* wsr = P.ws[src];
      __amdgpu_buffer_rsrc_t rs = mk_rsrc(wsr, WS_NEED), rd = mk_rsrc(P.out, OUT_BYTES);
      LAS char* mb = lds + wid * 32768;
      LAS char* sb = mb + 24576;
      const int fsub = lane < nsub ? k + DPL * lane : 0;
      const int ftile = (fsub >> 6) * 8 + ((fsub & 31) >> 2);
      const char* mfp = wsl + OFF_MFLAG + (src * 256 + ftile) * 4;
      const char* vfp = wsl + OFF_VFLAG + (src * 256 + ftile) * 4;
      int cntv = 0;
      unsigned pollv = 0;
      {
        u64 todo = all, ready = 0;
        const u64 t0 = NOW();
        DBG(bool first = true;)
        while (todo) {
          u64 avail = ready & todo;
          if (!avail) {
            ready = __ballot(ld_sys(mfp) == call);
            if (!(ready & todo) && give_up(wsl, call, t0)) { ready = ~0ull; bad = true; }
            continue;
          }
          DBG(if (first) STAMP(3); first = false;)
          int id[4]; u32x3 mv[4];
#pragma unroll
          for (int x = 0; x < 4; ++x) {
            id[x] = avail ? __builtin_ctzll(avail) : -1; avail &= avail - 1;
            mv[x] = __builtin_amdgcn_raw_buffer_load_b96(rs, lane * 12, id[x] >= 0 ? OFF_EXM + (unsigned)(k + DPL * id[x]) * 768u : WS_NEED, AUX_SYS);
          }
          todo &= ~ready | avail;
          pollv = ld_sys(todo ? mfp : vfp);
          WAIT_V0();
#pragma unroll
          for (int x = 0; x < 4; ++x) {
            if (id[x] >= 0) {
              *(LAS u32x3a4*)(mb + id[x] * 768 + lane * 12) = mv[x];
              const unsigned sc = scan_incl(__builtin_popcount(mv[x][0]) + __builtin_popcount(mv[x][1]) + __builtin_popcount(mv[x][2]));
              const int tot = __builtin_amdgcn_readlane((int)sc, 63); cntv = lane == id[x] ? tot : cntv;
            }
          }
          if (todo) ready |= __ballot(pollv == call);
        }
      }
      STAMP(4);
      {
        u64 todo = all, ready = 0; int cur = -1; unsigned curcnt = 0;
        u32x3 v[6] = {}; unsigned hv = 0;
        u32x4 q[6]; unsigned qoff = 0; bool pend = false;
        const u64 t0 = NOW();
        const int fr = lane & 15, fq = lane >> 4;
        const unsigned ovoff = (unsigned)fr * GN + fq * 16;
        for (;;) {
          ready |= __ballot(pollv == call);
          u32x3 o[6]; const unsigned ho = hv;
#pragma unroll
          for (int c = 0; c < 6; ++c) o[c] = v[c];
          const u64 avail = ready & todo;
          int nxt = -1; unsigned nxtcnt = 0;
          if (avail) {
            nxt = __builtin_ctzll(avail); todo &= ~(1ull << nxt);
            nxtcnt = (unsigned)__builtin_amdgcn_readlane(cntv, nxt);
            const unsigned jn = (unsigned)(k + DPL * nxt), so = OFF_EXV + jn * 4608u;
            __amdgpu_buffer_rsrc_t rv = mk_rsrc(wsr, so + ((nxtcnt + 15u) >> 4) * 12u);
            hv = __builtin_amdgcn_raw_buffer_load_b16(mk_rsrc(wsr, OFF_EXH + jn * 64u + 64u), lane * 2, OFF_EXH + jn * 64u, AUX_SYS);
#pragma unroll
            for (int c = 0; c < 6; ++c) v[c] = __builtin_amdgcn_raw_buffer_load_b96(rv, lane * 12 + c * 768, so, AUX_SYS);
          }
          if (todo & ~ready) pollv = ld_sys(vfp);
          FENCE();
          if (pend) {
#pragma unroll
            for (int m = 0; m < 6; ++m) ST128(q[m], rd, ovoff + m * 16 * GN, qoff, AUX_OUT);
            pend = false;
          }
          FENCE();
          if (cur >= 0) {
            const int j = k + DPL * cur;
            const u32x3 mk = *(const LAS u32x3a4*)(mb + cur * 768 + lane * 12);
            const unsigned nexc = (unsigned)uni((int)ho);
            const bool dense = nexc > 31u;
#pragma unroll
            for (int c = 0; c < 6; ++c) {
              if (c * 1024u < curcnt) {
                u32x4 d;
                d[0] = o[c][0] & 0x3f3f3f3fu; d[1] = o[c][1] & 0x3f3f3f3fu; d[2] = o[c][2] & 0x3f3f3f3fu;
                d[3] = ((o[c][0] >> 6) & 0x03030303u) | ((o[c][1] >> 4) & 0x0c0c0c0cu) | ((o[c][2] >> 2) & 0x30303030u);
                if (dense) {
                  const unsigned h = __builtin_amdgcn_raw_buffer_load_b16(rs, lane * 2 + c * 128, OFF_EXD + (unsigned)j * 768u, AUX_SYS);
                  const unsigned hb = (h & 0x0f0fu) | ((h & 0xf0f0u) << 12);
                  d[0] |= (hb << 6) & 0x40404040u; d[1] |= (hb << 5) & 0x40404040u; d[2] |= (hb << 4) & 0x40404040u; d[3] |= (hb << 3) & 0x40404040u;
                }
                *(LAS u32x4*)(sb + c * 1024 + lane * 16) = d;
              }
            }
            if (!dense && lane >= 1 && (unsigned)lane <= nexc)
              asm volatile("ds_or_b32 %0, %1" :: "v"((unsigned)(size_t)sb + (ho & 0x1ffcu)), "v"(0x40u << (8 * (ho & 3u))) : "memory");
            unsigned off[6];
            (void)piece_offsets(mk, off);
            qoff = (unsigned)((j >> 5) * 96) * GN + src * NLOC + (j & 31) * 64;
#pragma unroll
            for (int m = 0; m < 6; ++m) {
              unsigned a = off[m];
#pragma unroll
              for (int r = 0; r < 4; ++r) {
                const int kk = (m & 1) * 4 + r;
                const unsigned g = (kk >= 2 ? mk[m >> 1] >> (kk - 2) : mk[m >> 1] << (2 - kk)) & 0x04040404u;
                const u32x2 sv = *(const LAS u32x2a4*)(sb + (a & ~3u));
                const unsigned s1 = __builtin_amdgcn_alignbyte(sv[1], sv[0], a);
                a += __builtin_popcount(g);
                q[m][r] = __builtin_amdgcn_perm(0u, s1, g * 0x40404040u + (g ^ 0x04040404u));
              }
            }
            if (bad) q[0][0] = 0x80808080u;
            pend = true;
          } else if (nxt < 0) {
            if (!todo) break;
            if (give_up(wsl, call, t0)) { ready = ~0ull; bad = true; }
          }
          cur = nxt; curcnt = nxtcnt;
          WAIT_V0();
          asm volatile("" : "+v"(pollv), "+v"(hv), "+v"(v[0]), "+v"(v[1]), "+v"(v[2]), "+v"(v[3]), "+v"(v[4]), "+v"(v[5]));
        }
      }
      STAMP(6);
    }
  } else {
    const int g = b - GEMM0;
    if (START_SLEEP && g >= 80) __builtin_amdgcn_s_sleep(START_SLEEP);
    conv_unit(P, wsl, lds, g, wid, lane, prev);
    STAMP(1);
    conv_unit(P, wsl, lds, NGEMM + g, wid, lane, prev);
    STAMP(6);
    const unsigned call = (unsigned)uni((int)prev) + 1u;
    const int L = (g & 7) * 22 + (g >> 3);
    const int pm = (L % 88) >> 2, pn = (L / 88) * 4 + (L & 3);
    const int wr = wid >> 2, wc = wid & 3;
    volatile LAS int* ctl = (volatile LAS int*)(lds + LDS_CTL);
    __amdgpu_buffer_rsrc_t rl = mk_rsrc(wsl, WS_NEED);
    const unsigned wantA = call * (pm == 21 ? 2u : 6u), wantB = call * 8u;
    f32x4 acc[6][4];
#pragma unroll
    for (int m = 0; m < 6; ++m)
#pragma unroll
      for (int n = 0; n < 4; ++n) acc[m][n] = (f32x4){0.f, 0.f, 0.f, 0.f};
    const unsigned gvoff = lane * 16;
    const bool real = wid >= 3 || wid == 0 || pm != 21;
    const unsigned gbase = !real ? WS_NEED : wid < 3 ? OFF_STA + pm * 12288u + wid * 4096u : OFF_STB + pn * 32768u + (wid - 3) * 4096u;
    const unsigned gk = wid < 3 ? 524288u : 262144u, gh = wid < 3 ? 262144u : 16384u;
    const int gdst = wid * 4096;
    const int aoff = wr * 6144 + lane * 16, boff = 12288 + wc * 4096 + lane * 16;
    const unsigned pcopy = (unsigned)(size_t)(lds + LDS_CTL + 1536);
#define GL1(i) __builtin_amdgcn_raw_ptr_buffer_load_lds(rl, lds + sl * SLOT + gdst, 16, gvoff, so, (i) * 1024, 0)
#define GETE() const int e = uni(ev); const unsigned so = e >= 0 ? gbase + (unsigned)(e >> 1) * gk + (e & 1) * gh : WS_NEED; \
      rm |= e >= 0 ? 1u << NS : 0u; SCHED();
#define GL4() do { GL1(0); GL1(1); GL1(2); GL1(3); SCHED(); } while (0)
#define POLL() do { __builtin_amdgcn_raw_ptr_buffer_load_lds(rl, lds + LDS_CTL + 1536, 4, lane * 4, OFF_CNTA + pm * CSTR, 0, AUX_AG); \
        __builtin_amdgcn_raw_ptr_buffer_load_lds(rl, lds + LDS_CTL + 1792, 4, lane * 4, OFF_CNTB + pn * CSTR, 0, AUX_AG); } while (0)
#define RDALL(NA, NB) do { const LAS char* rb = lds + nx * SLOT; \
      _Pragma("unroll") for (int n = 0; n < 4; ++n) NB[n] = *(const LAS bf16x8*)(rb + boff + n * 1024); \
      _Pragma("unroll") for (int m = 0; m < 6; ++m) NA[m] = *(const LAS bf16x8*)(rb + aoff + m * 1024); SCHED(); } while (0)
#define MM(CA, CB) do { if (rm & 1u) { \
      _Pragma("unroll") for (int m = 0; m < 6; ++m) \
        _Pragma("unroll") for (int n = 0; n < 4; ++n) acc[m][n] = __builtin_amdgcn_mfma_f32_16x16x32_bf16(CB[n], CA[m], acc[m][n], 0, 0, 0); \
      ++ndone; } SCHED(); } while (0)
#define TS(i) DBG2({ u64 w_; asm volatile("s_memtime %0\n\ts_waitcnt lgkmcnt(0)" : "=s"(w_) :: "memory"); seg[i] += (unsigned)w_ - wl; wl = (unsigned)w_; SCHED(); })
#define STEP(G, par, CA, CB, NA, NB) { const int nx = sl + 1 == NS ? 0 : sl + 1; \
      if (G == 0) { const int ev = ctl[(par) ^ 1]; MM(CA, CB); TS(0) GETE() RDALL(NA, NB); GL4(); TS(2) WAIT_V(12); } \
      else if (G == 1) { const int ev = ctl[(par) ^ 1]; SCHED(); RDALL(NA, NB); GETE() GL4(); TS(2) MM(CA, CB); TS(0) WAIT_V(12); } \
      else { \
        rm |= last >= 0 ? 1u << NS : 0u; \
        if (rm == 0u && (ready & ~issued) == 0ull && ready != ~0ull) {        \
          if (ndone > 0 && ndone < FAST_AT) __builtin_amdgcn_s_sleep(IDLE_SLEEP); \
          POLL(); WAIT_V0(); cost = 0; \
          if ((++spins & 63) == 0 && give_up(wsl, call, t0)) ready = ~0ull; \
        } \
        unsigned pa, pb; \
        asm volatile("ds_read_b32 %0, %2\n\tds_read_b32 %1, %2 offset:256" : "=&v"(pa), "=&v"(pb) : "v"(pcopy + lane * 4) : "memory"); \
        SCHED(); RDALL(NA, NB); \
        asm volatile("s_waitcnt lgkmcnt(10)" : "+v"(pa), "+v"(pb) :: "memory"); \
        ready |= __builtin_amdgcn_ballot_w64(pa >= wantA) & __builtin_amdgcn_ballot_w64(pb >= wantB); \
        const u64 avail = ready & ~issued; \
        const bool dec = (last & 1) != 0;                   \
        const int kn = __builtin_ctzll(avail | (1ull << 63)); \
        const bool take = dec && avail != 0ull; \
        last = dec ? (take ? 2 * kn : -1) : last + 1; \
        issued |= take ? 1ull << kn : 0ull; \
        ctl[par] = last; \
        cost += (rm & 1u) ? 3 : 1; \
        const u64 left = ready & ~issued; \
        if (ready != ~0ull && (left & (left - 1)) == 0ull && cost >= (ndone < FAST_AT ? POLL_SLOW : POLL_COST)) { cost = 0; WAIT_V(12); POLL(); } \
        DBG(++nstep;) \
        SCHED(); TS(2) MM(CA, CB); TS(0) } \
      TS(3) \
      BAR(); sl = nx; rm >>= 1; TS(4) }
    bf16x8 xa[6], xb[4], ya[6], yb[4];
#pragma unroll
    for (int m = 0; m < 6; ++m) { xa[m] = (bf16x8)(__bf16)0.f; ya[m] = xa[m]; }
#pragma unroll
    for (int n = 0; n < 4; ++n) { xb[n] = (bf16x8)(__bf16)0.f; yb[n] = xb[n]; }
    int sl = 0, ndone = 0; unsigned rm = 0;
    DBG(int nstep = 0;)
    if (tid == 0) ctl[1] = -1;
    if (tid < 128) ctl[384 + tid] = 0;
    const u64 t0 = NOW();
    DBG(const u64 c0 = __builtin_amdgcn_s_memtime(); unsigned seg[5] = {}; unsigned wl = (unsigned)c0;)
    BAR();
    {
      u64 issued = 0, ready = 0; int last = -1, cost = 0; unsigned spins = 0;
      if (wid < 4) { do { STEP(0, 0, xa, xb, xa, xb) STEP(0, 1, xa, xb, xa, xb) } while (ndone != 128); }
      else if (wid < 7) { do { STEP(1, 0, xa, xb, ya, yb) STEP(1, 1, ya, yb, xa, xb) } while (ndone != 128); }
      else { do { STEP(2, 0, xa, xb, ya, yb) STEP(2, 1, ya, yb, xa, xb) } while (ndone != 128); }
    }
    DBG({ const u64 c1 = __builtin_amdgcn_s_memtime(); const u64 t1 = NOW(); STAMPV(12, (c1 - c0) * 1000 / (t1 - t0)); DBG2(STAMPV(13, seg[0] / 23); STAMPV(14, seg[1] / 23); STAMPV(15, seg[2] / 23); STAMPV(1, seg[3] / 23); STAMPV(6, seg[4] / 23);) })
    DBG(if (wid == 7) STAMPV(10, (u64)nstep * 50);)
    STAMP(2);
    if (pm == 21) {
#pragma unroll
      for (int m = 0; m < 6; ++m)
        if (wr == 1 || m >= 4) {
#pragma unroll
          for (int n = 0; n < 4; ++n) acc[m][n] = (f32x4){-1.f, -1.f, -1.f, -1.f};
        }
    }
    const int tile = pm * 8 + pn, sub = (pm * 2 + wr) * 32 + pn * 4 + wc;
    u32x3 mk;
#pragma unroll
    for (int D = 0; D < 3; ++D) {
      unsigned mm = 0;
#pragma unroll
      for (int n = 3; n >= 0; --n)
#pragma unroll
        for (int kk = 7; kk >= 0; --kk) mm = __builtin_amdgcn_alignbit(mm, __float_as_uint(acc[2 * D + (kk >> 2)][n][kk & 3]), 31);
      mk[D] = ~mm;
    }
    ST96(mk, rl, lane * 12, OFF_EXM + (unsigned)sub * 768u, AUX_SYS);
    float mx = 0.f;
#pragma unroll
    for (int m = 0; m < 6; ++m)
#pragma unroll
      for (int n = 0; n < 4; ++n) mx = fmaxf(fmaxf(mx, fmaxf(acc[m][n][0], acc[m][n][1])), fmaxf(acc[m][n][2], acc[m][n][3]));
#pragma unroll
    for (int o = 32; o >= 1; o >>= 1) mx = fmaxf(mx, __shfl_xor(mx, o));
    volatile LAS float* red = (volatile LAS float*)(lds + LDS_CTL + 64);
    if (lane == 0) red[wid] = mx;
    WAIT_V0(); BAR();
    if (wid == 1 && lane < NR && lane != rank) st_sys(P.ws[lane] + OFF_MFLAG + (rank * 256 + tile) * 4, call);
    STAMP(3);
    if (wid == 0) {
      float v = red[lane & 7];
#pragma unroll
      for (int o = 4; o >= 1; o >>= 1) v = fmaxf(v, __shfl_xor(v, o));
      unsigned last = 0;
      if (lane == 0) {
        (void)__hip_atomic_fetch_max((GAS u64*)(wsl + OFF_AMAXL), ((u64)call << 32) | __float_as_uint(v), __ATOMIC_RELAXED, __HIP_MEMORY_SCOPE_AGENT);
        WAIT_V0();
        last = __hip_atomic_fetch_add((GAS unsigned*)(wsl + OFF_AMAXT), 1u, __ATOMIC_RELAXED, __HIP_MEMORY_SCOPE_AGENT) + 1u == call * (unsigned)NGEMM;
      }
      if (uni((int)last)) {
        const u64 gm = __hip_atomic_load((GAS u64*)(wsl + OFF_AMAXL), __ATOMIC_RELAXED, __HIP_MEMORY_SCOPE_AGENT);
        if (lane < NR) st_sys64(P.ws[lane] + OFF_AMAXG + rank * 128, gm);
      }
      const u64 t1 = NOW(); u64 gv;
      for (;;) {
        gv = ld_sys64(wsl + OFF_AMAXG + (lane & 7) * 128);
        if (__all((unsigned)(gv >> 32) == call)) break;
        if (give_up(wsl, call, t1)) break;
      }
      float am = __uint_as_float((unsigned)gv);
#pragma unroll
      for (int o = 4; o >= 1; o >>= 1) am = fmaxf(am, __shfl_xor(am, o));
      const unsigned tm = ld_ag(wsl + OFF_TMO);
      if (lane == 0) { red[8] = am; red[9] = (tm == call) ? 1.f : 0.f; }
    }
    unsigned off[6];
    const unsigned cnt = piece_offsets(mk, off);
    BAR();
    STAMP(4);
    const float amax = red[8]; const bool bad = red[9] != 0.f;
    const float inv = amax > 0.f ? 127.f / amax : 0.f;
    u32x4 q[6];
#pragma unroll
    for (int m = 0; m < 6; ++m) {
      unsigned a = wid * 8192 + off[m];
#pragma unroll
      for (int r = 0; r < 4; ++r) {
        unsigned d = 0;
#pragma unroll
        for (int n = 0; n < 4; ++n) d = __builtin_amdgcn_cvt_pk_u8_f32(__builtin_rintf(acc[m][n][r] * inv), n, d);
        q[m][r] = d;
        const unsigned d8 = d >> 8;
#pragma unroll
        for (int n = 0; n < 4; ++n) {
          const int sg = (int)__float_as_uint(acc[m][n][r]) >> 31;
          lds[a | ((unsigned)sg & 0x10000u)] = (char)(((n & 1) ? d8 : d) >> ((n & 2) * 8));
          a += 1 + sg;
        }
      }
    }
    {
      LAS unsigned short* hd = (LAS unsigned short*)(lds + LDS_CTL + 256 + wid * 128);
      const unsigned so = OFF_EXV + (unsigned)sub * 4608u;
      unsigned nexc = 0;
#pragma unroll 1
      for (unsigned c = 0; c * 1024u < cnt; ++c) {
        const u32x4 d = *(const LAS u32x4*)(lds + wid * 8192 + c * 1024 + lane * 16);
        u32x3 o;
        o[0] = (d[0] & 0x3f3f3f3fu) | ((d[3] << 6) & 0xc0c0c0c0u);
        o[1] = (d[1] & 0x3f3f3f3fu) | ((d[3] << 4) & 0xc0c0c0c0u);
        o[2] = (d[2] & 0x3f3f3f3fu) | ((d[3] << 2) & 0xc0c0c0c0u);
        ST96(o, rl, lane * 12 + c * 768, so, AUX_SYS);
        unsigned hb = ((d[0] >> 6) & 0x01010101u) | ((d[1] >> 5) & 0x02020202u) | ((d[2] >> 4) & 0x04040404u) | ((d[3] >> 3) & 0x08080808u);
        __builtin_amdgcn_raw_buffer_store_b16((unsigned short)((hb & 0x0f0fu) | ((hb >> 12) & 0xf0f0u)), rl, lane * 2 + c * 128, OFF_EXD + (unsigned)sub * 768u, AUX_SYS);
        const unsigned pos0 = c * 1024 + lane * 16;
        if (pos0 >= cnt) hb = 0;
        for (;;) {
          const u64 bal = __ballot(hb != 0);
          if (!bal || nexc > 31u) break;
          const unsigned sl = nexc + __builtin_amdgcn_mbcnt_hi((unsigned)(bal >> 32), __builtin_amdgcn_mbcnt_lo((unsigned)bal, 0u));
          if (hb != 0 && sl < 31u) { const unsigned bi = __builtin_ctz(hb); hd[1 + sl] = (unsigned short)(pos0 + 4 * (bi & 3) + (bi >> 3)); }
          hb &= hb - 1;
          nexc += __builtin_popcountll(bal);
        }
      }
      if (lane == 0) hd[0] = (unsigned short)(nexc > 31u ? 0xffffu : nexc);
      if (lane < 16) __builtin_amdgcn_raw_buffer_store_b32(((LAS unsigned*)hd)[lane], rl, lane * 4, OFF_EXH + (unsigned)sub * 64u, AUX_SYS);
    }
    WAIT_V0(); BAR();
    if (tid < NR && tid != rank) st_sys(P.ws[tid] + OFF_VFLAG + (rank * 256 + tile) * 4, call);
    STAMP(5);
    {
      __amdgpu_buffer_rsrc_t rd = mk_rsrc(P.out, OUT_BYTES);
      const unsigned ovoff = (unsigned)(wr * 96 + (lane & 15)) * GN + wc * 64 + (lane >> 4) * 16;
      const unsigned osoff = (unsigned)(pm * 192) * GN + rank * NLOC + pn * 256;
#pragma unroll
      for (int m = 0; m < 6; ++m) {
        if (bad) q[m][0] = 0x80808080u;
        ST128(q[m], rd, ovoff + m * 16 * GN, osoff, AUX_OUT);
      }
    }
  }
  __syncthreads();
  if (tid == 0) st_ag(wsl + OFF_CALL + b * 64, prev + 1u);
}
}

extern "C" void kernel_launch(int rank, int nranks, ncclComm_t comm, hipStream_t stream,
                              void* const* d_in, const int* in_sizes, int n_in,
                              void* d_out, int out_size, const int* in_dtypes, int out_dtype,
                              void* const* d_out_all, void* const* d_ws_all, size_t ws_bytes) {
  if (nranks != NR || (unsigned)rank >= (unsigned)NR || ws_bytes < WS_NEED || out_size != GM * GN) { fprintf(stderr, "unexpected configuration\n"); return; }
  Params P;
  P.x = (const float*)d_in[0]; P.w = (const float*)d_in[1]; P.out = (char*)d_out;
  for (int p = 0; p < NR; ++p) P.ws[p] = (char*)d_ws_all[p];
  P.rank = rank;
  hipLaunchKernelGGL(fused_kernel, dim3(NWG), dim3(512), 0, stream, P);
}
```

```cpp
#include <hip/hip_runtime.h>
#include <rccl/rccl.h>
#include <cstdio>

#define LAS __attribute__((address_space(3)))
#define GAS __attribute__((address_space(1)))
typedef unsigned u32x4 __attribute__((ext_vector_type(4)));
typedef unsigned u32x3 __attribute__((ext_vector_type(3)));
typedef unsigned u32x2 __attribute__((ext_vector_type(2)));
typedef u32x2 u32x2a4 __attribute__((aligned(4)));
typedef u32x3 u32x3a4 __attribute__((aligned(4)));
typedef float f32x4 __attribute__((ext_vector_type(4)));
typedef float f32x2 __attribute__((ext_vector_type(2)));
typedef __bf16 bf16x2 __attribute__((ext_vector_type(2)));
typedef __bf16 bf16x8 __attribute__((ext_vector_type(8)));
typedef unsigned long long u64;

namespace {
constexpr int NR = 8;
constexpr int GM = 4096, GN = 16384, KSH = 512, NLOC = 2048;
constexpr int NWG = 256, NLINKWG = 56, NCONVWG = 24, GEMM0 = 80, NGEMM = 176;
constexpr int NSUB = 1376;
constexpr int PWV = 3, XW = 6, WPL = PWV * 8 + XW;
constexpr int DPL = 40, DPG = 36;
constexpr int NPUSH = 64;
constexpr int TASKS_IN = 1536;
constexpr int PMP = 14;
constexpr int TASKS_ALL = TASKS_IN + NSUB / 4 - 16 * PMP;
constexpr int TR0 = 3, TUM = 6;
constexpr int NUNIT = 640;
constexpr unsigned MiB = 1u << 20;
constexpr unsigned OFF_CALL = 0;
constexpr unsigned OFF_TMO = 16384;
constexpr unsigned CSTR = 4352;
constexpr unsigned OFF_UFLAG = 65536;
constexpr unsigned OFF_MFLAG = OFF_UFLAG + 8 * CSTR;
constexpr unsigned OFF_VFLAG = OFF_MFLAG + 8 * CSTR;
constexpr unsigned OFF_CNTA = OFF_VFLAG + 8 * CSTR;
constexpr unsigned OFF_CNTB = OFF_CNTA + 22 * CSTR;
constexpr unsigned OFF_TICK = OFF_CNTB + 8 * CSTR;
constexpr unsigned OFF_VTICK = OFF_TICK + 8 * CSTR;
constexpr unsigned OFF_MRC = OFF_VTICK + 8 * CSTR;
constexpr unsigned OFF_AMX = OFF_MRC + 8 * CSTR;
constexpr unsigned OFF_AMF = OFF_AMX + 8 * CSTR;
constexpr unsigned OFF_STA = 1 * MiB;
constexpr unsigned OFF_STB = 33 * MiB;
constexpr unsigned OFF_XLE = 49 * MiB, XES = 4352;
constexpr unsigned OFF_VT = 72 * MiB;
constexpr unsigned OFF_OVF = 74 * MiB;
constexpr unsigned OFF_EXM = 75 * MiB;
constexpr unsigned OFF_EXV = 77 * MiB, XVS = 4352;
constexpr unsigned OFF_EXD = 83 * MiB;
constexpr unsigned OFF_LM = 86 * MiB;
constexpr unsigned WS_NEED = 95 * MiB;
constexpr unsigned OUT_BYTES = (unsigned)GM * GN;
constexpr u64 DEADLINE = 50000000ull;
constexpr int SLOT = 28672, NS = 3;
constexpr int LDS_CTL = 8 * 18432 + 2048;
constexpr int LDS_BYTES = LDS_CTL + 2048;
constexpr int AUX_SYS = 17, AUX_AG = 16, AUX_NT = 2, AUX_OUT = 18;
constexpr int POLL_COST = 4, POLL_SLOW = 16, FAST_AT = 104, BTH = 7;
constexpr int PT0 = 0x3041, PT1 = 0x924;

__device__ const __attribute__((aligned(16))) unsigned short ORD[7 * 768] = {
0,1,2,3,4,5,6,7,8,9,10,11,12,13,14,15,16,17,18,19,20,21,22,23,24,25,26,27,28,29,30,31,
32,33,34,35,36,37,38,39,40,41,42,43,47,48,49,56,57,58,44,45,46,59,60,61,53,54,55,50,51,52,62,63,
64,83,84,85,68,69,70,92,93,94,65,66,67,89,90,91,71,72,73,74,75,76,77,78,79,86,87,88,95,80,81,82,
128,129,130,131,132,133,134,135,136,137,138,160,161,162,139,140,141,142,163,164,165,143,144,145,146,147,148,149,150,151,152,153,
154,155,166,167,168,156,157,158,169,170,171,159,256,257,258,259,175,176,177,260,261,262,263,264,265,184,185,186,266,172,173,174,
288,289,290,267,268,269,187,188,189,270,291,292,293,271,272,273,274,275,276,181,182,183,277,278,279,280,281,282,178,179,180,283,
294,295,296,284,285,286,297,298,299,287,384,385,386,387,303,304,305,190,191,192,388,389,211,212,213,390,196,197,198,391,220,221,
222,392,393,312,313,314,394,300,301,302,416,417,418,395,396,193,194,195,397,315,316,317,398,419,420,421,399,217,218,219,400,401,
402,403,199,200,201,202,203,204,404,309,310,311,405,406,407,408,205,206,207,214,215,216,409,410,306,307,308,411,422,423,424,412,
413,414,223,425,426,427,415,208,209,210,512,513,514,515,431,432,433,318,319,320,516,517,339,340,341,518,324,325,326,519,348,349,
350,520,521,440,441,442,522,428,429,430,544,545,546,523,524,321,322,323,525,443,444,445,526,547,548,549,527,345,346,347,528,529,
530,531,327,328,329,330,331,332,532,437,438,439,533,534,535,536,333,334,335,342,343,344,537,538,434,435,436,539,550,551,552,540,
541,542,351,553,554,555,543,336,337,338,640,641,642,559,560,561,643,446,447,448,644,645,467,468,469,646,452,453,454,647,476,477,
478,648,649,568,569,570,650,556,557,558,672,673,674,651,652,449,450,451,653,571,572,573,654,675,676,677,655,473,474,475,656,657,
658,455,456,457,659,458,459,460,660,565,566,567,661,662,663,664,461,462,463,470,471,472,665,666,562,563,564,667,668,678,679,680,
669,670,479,464,465,466,671,681,682,683,768,769,770,771,687,688,689,574,575,576,772,773,595,596,597,774,580,581,582,775,604,605,
606,776,777,778,800,801,802,779,684,685,686,780,696,697,698,577,578,579,781,782,803,804,805,783,601,602,603,699,700,701,784,785,
786,787,583,584,585,586,587,588,788,789,790,693,694,695,791,792,589,590,591,598,599,600,793,794,795,796,806,807,808,690,691,692,
797,798,607,799,592,593,594,809,810,811,896,897,898,899,815,816,817,900,901,902,702,703,704,903,904,905,723,724,725,708,709,710,
906,928,929,930,812,813,814,907,824,825,826,732,733,734,908,909,931,932,933,910,827,828,829,911,705,706,707,912,913,914,729,730,
731,915,916,917,821,822,823,918,711,712,713,919,714,715,716,920,921,922,934,935,936,923,818,819,820,924,717,718,719,726,727,728,
925,937,938,939,926,927,735,720,721,722,943,944,945,830,831,832,851,852,853,952,953,954,836,837,838,940,941,942,860,861,862,955,
956,957,833,834,835,857,858,859,949,950,951,839,840,841,842,843,844,946,947,948,845,846,847,854,855,856,863,848,849,850,958,959,
960,979,980,981,988,989,990,964,965,966,961,962,963,985,986,987,967,968,969,970,971,972,973,974,975,982,983,984,991,976,977,978,
28,29,30,31,0,1,2,3,4,5,6,7,8,9,10,11,12,13,14,15,16,17,18,19,20,21,22,23,24,25,26,27,
32,33,34,35,36,37,38,39,40,50,51,52,41,42,43,47,48,49,56,57,58,44,45,46,59,60,61,53,54,55,74,75,
76,95,80,81,82,62,63,64,83,84,85,68,69,70,92,93,94,65,66,67,89,90,91,71,72,73,77,78,79,86,87,88,
156,157,158,159,128,129,130,131,132,133,134,160,161,162,135,136,137,138,163,164,165,139,140,141,142,143,144,145,146,147,148,149,
150,151,166,167,168,152,153,178,179,180,154,169,170,171,155,284,285,286,287,175,176,177,256,257,258,259,260,261,184,185,186,262,
172,173,174,288,289,290,263,264,265,187,188,189,266,291,292,293,267,268,269,270,271,272,181,182,183,273,274,202,203,204,275,276,
277,278,279,294,295,296,280,281,306,307,308,282,297,298,299,283,412,223,208,209,210,413,414,415,303,304,305,190,191,192,384,385,
211,212,213,386,196,197,198,387,220,221,222,388,389,312,313,314,390,300,301,302,416,417,418,391,392,193,194,195,393,315,316,317,
394,419,420,421,395,217,218,219,396,397,398,399,199,200,201,400,309,310,311,401,402,330,331,332,403,404,205,206,207,214,215,216,
405,406,407,422,423,424,408,409,434,435,436,410,425,426,427,411,540,351,336,337,338,541,542,543,431,432,433,318,319,320,512,513,
339,340,341,514,324,325,326,515,348,349,350,516,517,440,441,442,518,428,429,430,544,545,546,519,520,321,322,323,521,443,444,445,
522,547,548,549,523,345,346,347,524,525,526,527,327,328,329,528,437,438,439,529,530,458,459,460,531,532,333,334,335,342,343,344,
533,534,535,550,551,552,536,537,562,563,564,538,553,554,555,539,668,479,464,465,466,669,670,559,560,561,671,446,447,448,640,641,
467,468,469,642,452,453,454,643,476,477,478,644,645,568,569,570,646,556,557,558,672,673,674,647,648,449,450,451,649,571,572,573,
650,675,676,677,651,473,474,475,652,653,654,455,456,457,655,656,565,566,567,657,658,586,587,588,659,660,461,462,463,470,471,472,
661,662,663,664,678,679,680,665,666,667,690,691,692,681,682,683,796,607,592,593,594,797,798,799,687,688,689,574,575,576,768,769,
595,596,597,770,580,581,582,771,604,605,606,772,773,774,800,801,802,775,684,685,686,776,696,697,698,577,578,579,777,778,803,804,
805,779,601,602,603,699,700,701,780,781,782,783,583,584,585,784,785,786,693,694,695,787,788,589,590,591,598,599,600,789,790,714,
715,716,791,792,806,807,808,793,794,818,819,820,795,809,810,811,924,925,926,927,815,816,817,896,735,720,721,722,897,898,702,703,
704,899,900,901,723,724,725,708,709,710,902,928,929,930,812,813,814,903,824,825,826,732,733,734,904,905,931,932,933,906,827,828,
829,907,705,706,707,908,909,910,729,730,731,911,912,913,821,822,823,914,711,712,713,915,916,842,843,844,917,918,934,935,936,919,
946,947,948,920,717,718,719,726,727,728,921,937,938,939,922,923,943,944,945,863,848,849,850,830,831,832,851,852,853,952,953,954,
836,837,838,940,941,942,860,861,862,955,956,957,833,834,835,857,858,859,949,950,951,970,971,972,839,840,841,845,846,847,854,855,
856,991,976,977,978,958,959,960,979,980,981,988,989,990,964,965,966,961,962,963,985,986,987,967,968,969,973,974,975,982,983,984,
24,25,26,27,28,29,30,31,0,1,2,3,4,5,6,7,8,9,10,11,12,13,14,15,16,17,18,19,20,21,22,23,
32,33,34,35,36,37,38,39,40,50,51,52,41,42,43,47,48,49,56,57,58,44,45,46,59,60,61,65,66,67,53,54,
55,71,72,73,74,75,76,77,78,79,86,87,88,95,80,81,82,62,63,64,83,84,85,68,69,70,92,93,94,89,90,91,
152,153,154,155,156,157,158,159,128,129,130,160,161,162,131,132,133,134,163,164,165,135,136,137,138,139,140,141,142,143,144,145,
146,147,166,167,168,148,149,178,179,180,150,169,170,171,151,280,281,282,283,175,176,177,284,285,286,287,256,257,184,185,186,258,
172,173,174,288,289,290,259,260,261,187,188,189,262,291,292,293,263,193,194,195,264,265,266,267,268,181,182,183,269,199,200,201,
270,202,203,204,271,272,273,274,275,205,206,207,214,215,216,294,295,296,276,277,306,307,308,278,297,298,299,279,408,223,208,209,
210,409,410,411,303,304,305,190,191,192,412,413,211,212,213,414,196,197,198,415,220,221,222,384,385,312,313,314,386,300,301,302,
416,417,418,387,388,389,315,316,317,390,419,420,421,391,217,218,219,321,322,323,392,393,394,395,396,309,310,311,397,327,328,329,
398,330,331,332,399,400,401,402,403,333,334,335,342,343,344,422,423,424,404,405,434,435,436,406,425,426,427,407,536,351,336,337,
338,537,538,539,431,432,433,318,319,320,540,541,339,340,341,542,324,325,326,543,348,349,350,512,513,440,441,442,514,428,429,430,
544,545,546,515,516,517,443,444,445,518,547,548,549,519,345,346,347,449,450,451,520,521,522,523,524,437,438,439,525,455,456,457,
526,458,459,460,527,528,529,530,531,461,462,463,470,471,472,550,551,552,532,533,562,563,564,534,553,554,555,535,664,479,464,465,
466,665,666,559,560,561,667,446,447,448,668,669,467,468,469,670,452,453,454,671,476,477,478,640,641,568,569,570,642,556,557,558,
672,673,674,643,644,645,571,572,573,646,675,676,677,647,473,474,475,577,578,579,648,649,650,651,652,565,566,567,653,583,584,585,
654,586,587,588,655,656,657,658,659,589,590,591,598,599,600,660,678,679,680,661,662,663,690,691,692,681,682,683,792,607,592,593,
594,793,794,795,687,688,689,574,575,576,796,797,595,596,597,798,580,581,582,799,604,605,606,768,769,770,800,801,802,771,684,685,
686,772,696,697,698,773,774,803,804,805,775,601,602,603,699,700,701,776,777,778,705,706,707,779,780,781,782,693,694,695,783,784,
785,711,712,713,786,714,715,716,787,788,806,807,808,789,790,818,819,820,791,717,718,719,809,810,811,726,727,728,920,921,922,923,
815,816,817,924,735,720,721,722,925,926,702,703,704,927,896,897,723,724,725,708,709,710,898,928,929,930,812,813,814,899,824,825,
826,732,733,734,900,901,931,932,933,902,827,828,829,903,904,905,833,834,835,906,729,730,731,907,908,909,821,822,823,910,911,839,
840,841,912,842,843,844,913,914,934,935,936,915,946,947,948,916,917,845,846,847,854,855,856,937,938,939,918,919,943,944,945,863,
848,849,850,830,831,832,851,852,853,952,953,954,836,837,838,940,941,942,860,861,862,955,956,957,961,962,963,857,858,859,949,950,
951,967,968,969,970,971,972,973,974,975,982,983,984,991,976,977,978,958,959,960,979,980,981,988,989,990,964,965,966,985,986,987,
20,21,22,23,24,25,26,27,28,29,30,31,0,1,2,3,4,5,6,7,8,9,10,11,12,13,14,15,16,17,18,19,
47,48,49,32,33,34,35,36,37,38,39,40,50,51,52,41,42,43,62,63,64,68,69,70,56,57,58,44,45,46,59,60,
61,65,66,67,89,90,91,53,54,55,71,72,73,74,75,76,77,78,79,86,87,88,95,80,81,82,83,84,85,92,93,94,
148,149,150,151,152,153,154,155,156,157,158,175,176,177,159,128,129,130,160,161,162,131,132,133,134,163,164,165,135,136,137,138,
139,140,141,142,143,144,145,146,147,166,167,168,276,277,178,179,180,278,169,170,171,279,280,281,282,283,284,285,286,303,304,305,
287,190,191,192,256,257,258,196,197,198,184,185,186,172,173,174,288,289,290,259,260,261,187,188,189,262,291,292,293,263,264,193,
194,195,265,266,217,218,219,267,268,181,182,183,269,270,199,200,201,271,202,203,204,272,273,274,275,205,206,207,214,215,216,294,
295,296,404,405,306,307,308,406,297,298,299,407,408,223,409,208,209,210,410,411,412,413,211,212,213,414,431,432,433,415,318,319,
320,220,221,222,384,385,386,324,325,326,312,313,314,300,301,302,416,417,418,387,388,389,315,316,317,390,419,420,421,391,392,321,
322,323,393,394,345,346,347,395,396,309,310,311,397,398,327,328,329,399,330,331,332,400,401,402,403,333,334,335,342,343,344,422,
423,424,532,533,434,435,436,534,425,426,427,535,536,351,537,336,337,338,538,539,540,541,339,340,341,542,559,560,561,543,446,447,
448,348,349,350,512,513,514,452,453,454,440,441,442,428,429,430,544,545,546,515,516,517,443,444,445,518,547,548,549,519,520,449,
450,451,521,522,473,474,475,523,524,437,438,439,525,526,455,456,457,527,458,459,460,528,529,530,531,461,462,463,470,471,472,550,
551,552,660,562,563,564,661,553,554,555,662,663,479,664,464,465,466,665,666,667,668,467,468,469,669,670,687,688,689,574,575,576,
671,476,477,478,640,641,580,581,582,568,569,570,556,557,558,672,673,674,642,643,644,645,571,572,573,646,675,676,677,647,577,578,
579,648,649,601,602,603,650,651,565,566,567,652,653,583,584,585,654,586,587,588,655,656,657,658,659,589,590,591,598,599,600,678,
679,680,788,789,790,690,691,692,681,682,683,791,792,607,592,593,594,793,794,795,796,797,595,596,597,798,815,816,817,799,604,605,
606,768,769,702,703,704,770,800,801,802,771,684,685,686,772,696,697,698,708,709,710,773,774,803,804,805,775,699,700,701,776,777,
778,705,706,707,779,780,781,729,730,731,782,693,694,695,783,784,785,711,712,713,786,714,715,716,787,916,806,807,808,917,918,818,
819,820,919,717,718,719,809,810,811,726,727,728,920,921,922,923,924,925,735,720,721,722,926,943,944,945,927,896,897,830,831,832,
898,723,724,725,899,928,929,930,900,812,813,814,824,825,826,836,837,838,732,733,734,901,902,903,931,932,933,827,828,829,904,905,
906,833,834,835,907,908,857,858,859,909,910,821,822,823,911,912,839,840,841,913,842,843,844,914,915,934,935,936,946,947,948,845,
846,847,854,855,856,937,938,939,863,848,849,850,958,959,960,964,965,966,851,852,853,952,953,954,940,941,942,860,861,862,955,956,
957,961,962,963,985,986,987,949,950,951,967,968,969,970,971,972,973,974,975,982,983,984,991,976,977,978,979,980,981,988,989,990,
16,17,18,19,20,21,22,23,24,25,26,27,28,29,30,31,0,1,2,3,4,5,6,7,8,9,10,11,12,13,14,15,
47,48,49,32,33,34,44,45,46,35,36,37,53,54,55,38,39,40,50,51,52,41,42,43,62,63,64,68,69,70,56,57,
58,59,60,61,65,66,67,89,90,91,71,72,73,74,75,76,77,78,79,86,87,88,95,80,81,82,83,84,85,92,93,94,
144,145,146,147,148,149,150,151,152,153,154,175,176,177,155,156,157,158,160,161,162,159,128,129,172,173,174,130,163,164,165,131,
132,133,134,135,136,137,138,139,181,182,183,140,141,142,143,166,167,168,272,273,178,179,180,274,169,170,171,275,276,277,278,279,
280,281,282,303,304,305,283,190,191,192,284,285,286,196,197,198,184,185,186,288,289,290,287,256,257,300,301,302,187,188,189,258,
291,292,293,259,260,193,194,195,261,262,217,218,219,263,264,265,266,199,200,201,267,202,203,204,309,310,311,268,269,270,271,205,
206,207,214,215,216,294,295,296,400,401,306,307,308,402,297,298,299,403,404,223,405,208,209,210,406,407,408,409,211,212,213,410,
431,432,433,411,318,319,320,220,221,222,412,413,414,324,325,326,312,313,314,416,417,418,415,384,385,428,429,430,315,316,317,386,
419,420,421,387,388,321,322,323,389,390,345,346,347,391,392,393,394,327,328,329,395,330,331,332,437,438,439,396,397,398,399,333,
334,335,342,343,344,422,423,424,528,529,434,435,436,530,425,426,427,531,532,351,533,336,337,338,534,535,536,537,339,340,341,538,
559,560,561,539,446,447,448,348,349,350,540,541,542,452,453,454,440,441,442,544,545,546,543,512,513,556,557,558,443,444,445,514,
547,548,549,515,516,449,450,451,517,518,473,474,475,519,520,521,522,455,456,457,523,458,459,460,565,566,567,524,525,526,527,461,
462,463,470,471,472,550,551,552,656,562,563,564,657,553,554,555,658,659,479,660,464,465,466,661,662,663,664,467,468,469,665,666,
687,688,689,574,575,576,667,476,477,478,668,669,580,581,582,568,569,570,672,673,674,670,671,640,641,571,572,573,642,684,685,686,
675,676,677,643,577,578,579,644,645,601,602,603,646,647,648,649,583,584,585,650,586,587,588,651,652,693,694,695,653,654,655,589,
590,591,598,599,600,678,679,680,784,785,786,690,691,692,681,682,683,787,788,607,592,593,594,789,790,791,792,793,595,596,597,794,
815,816,817,795,604,605,606,796,797,702,703,704,798,800,801,802,799,768,696,697,698,708,709,710,769,812,813,814,770,803,804,805,
771,699,700,701,772,773,774,705,706,707,775,776,777,729,730,731,778,779,821,822,823,780,781,711,712,713,782,714,715,716,783,912,
806,807,808,913,914,818,819,820,915,717,718,719,809,810,811,726,727,728,916,917,918,919,920,921,735,720,721,722,922,943,944,945,
923,924,925,830,831,832,926,723,724,725,927,928,929,930,896,824,825,826,836,837,838,732,733,734,940,941,942,897,898,899,931,932,
933,827,828,829,900,901,902,833,834,835,903,904,857,858,859,905,949,950,951,906,907,908,839,840,841,909,842,843,844,910,911,934,
935,936,946,947,948,845,846,847,854,855,856,937,938,939,863,848,849,850,958,959,960,964,965,966,851,852,853,952,953,954,860,861,
862,955,956,957,961,962,963,985,986,987,967,968,969,970,971,972,973,974,975,982,983,984,991,976,977,978,979,980,981,988,989,990,
12,13,14,15,16,17,18,19,20,21,22,23,24,25,26,27,28,29,30,31,0,1,2,3,4,5,6,7,8,9,10,11,
38,39,40,41,42,43,47,48,49,32,33,34,44,45,46,35,36,37,53,54,55,50,51,52,62,63,64,68,69,70,56,57,
58,59,60,61,65,66,67,89,90,91,71,72,73,74,75,76,77,78,79,86,87,88,95,80,81,82,83,84,85,92,93,94,
140,141,142,143,144,145,146,147,148,149,150,166,167,168,151,152,153,169,170,171,154,155,156,157,158,175,176,177,159,128,129,130,
160,161,162,131,132,133,172,173,174,134,163,164,165,135,136,137,138,139,268,269,270,181,182,183,271,272,273,274,275,276,277,178,
179,180,278,294,295,296,279,280,281,297,298,299,282,283,284,285,286,303,304,305,287,190,191,192,256,257,258,196,197,198,184,185,
186,288,289,290,259,260,261,300,301,302,262,187,188,189,291,292,293,263,264,193,194,195,265,266,217,218,219,267,396,397,199,200,
201,398,202,203,204,309,310,311,399,400,401,402,403,205,206,207,214,215,216,404,405,306,307,308,406,422,423,424,407,408,409,223,
425,426,427,208,209,210,410,411,412,413,414,211,212,213,431,432,433,415,318,319,320,384,220,221,222,385,386,324,325,326,312,313,
314,416,417,418,387,388,389,428,429,430,390,315,316,317,419,420,421,391,392,321,322,323,393,394,345,346,347,395,524,525,327,328,
329,526,330,331,332,437,438,439,527,528,529,530,531,333,334,335,342,343,344,532,533,434,435,436,534,550,551,552,535,536,537,351,
553,554,555,336,337,338,538,539,540,541,542,339,340,341,559,560,561,543,446,447,448,512,348,349,350,513,514,452,453,454,440,441,
442,544,545,546,515,516,517,556,557,558,518,443,444,445,547,548,549,519,520,449,450,451,521,522,473,474,475,523,652,455,456,457,
653,458,459,460,565,566,567,654,655,656,657,658,461,462,463,470,471,472,659,660,562,563,564,661,662,678,679,680,663,664,479,464,
465,466,665,681,682,683,666,667,668,669,467,468,469,670,687,688,689,574,575,576,671,476,477,478,640,641,580,581,582,568,569,570,
642,672,673,674,643,644,645,571,572,573,646,684,685,686,675,676,677,647,577,578,579,648,649,650,601,602,603,651,780,781,583,584,
585,782,586,587,588,783,784,693,694,695,785,786,589,590,591,598,599,600,787,788,789,790,806,807,808,690,691,692,791,792,607,793,
592,593,594,809,810,811,794,795,796,797,595,596,597,798,815,816,817,799,604,605,606,768,769,702,703,704,770,800,801,802,771,772,
696,697,698,708,709,710,773,774,812,813,814,803,804,805,775,776,699,700,701,777,778,779,705,706,707,908,909,910,729,730,731,911,
912,821,822,823,913,711,712,713,914,714,715,716,915,916,917,918,934,935,936,919,818,819,820,717,718,719,726,727,728,920,921,937,
938,939,922,923,924,925,735,926,720,721,722,943,944,945,927,896,897,830,831,832,898,723,724,725,899,928,929,930,900,824,825,826,
836,837,838,732,733,734,940,941,942,901,902,903,931,932,933,904,827,828,829,905,906,833,834,835,907,857,858,859,949,950,951,839,
840,841,842,843,844,946,947,948,845,846,847,854,855,856,863,848,849,850,958,959,960,964,965,966,851,852,853,952,953,954,860,861,
862,955,956,957,961,962,963,985,986,987,967,968,969,970,971,972,973,974,975,982,983,984,991,976,977,978,979,980,981,988,989,990,
8,9,10,11,12,13,14,15,16,17,18,19,20,21,22,23,24,25,26,27,28,29,30,31,0,1,2,3,4,5,6,7,
32,33,34,35,36,37,38,39,40,41,42,43,47,48,49,44,45,46,53,54,55,50,51,52,62,63,64,68,69,70,56,57,
58,92,93,94,59,60,61,65,66,67,89,90,91,71,72,73,74,75,76,77,78,79,86,87,88,95,80,81,82,83,84,85,
136,137,138,139,140,141,142,143,144,145,146,160,161,162,147,148,149,150,163,164,165,151,152,153,154,155,156,157,158,159,128,129,
130,131,166,167,168,132,133,134,169,170,171,135,264,265,266,267,175,176,177,268,269,270,271,272,273,274,172,173,174,288,289,290,
275,276,277,278,291,292,293,279,280,281,282,283,284,181,182,183,285,286,287,256,257,258,178,179,180,259,294,295,296,260,261,262,
297,298,299,263,392,393,394,395,303,304,305,190,191,192,396,397,398,196,197,198,399,184,185,186,220,221,222,400,401,402,300,301,
302,416,417,418,187,188,189,403,404,193,194,195,405,406,419,420,421,407,217,218,219,408,409,410,411,199,200,201,202,203,204,412,
309,310,311,413,414,415,384,205,206,207,214,215,216,385,386,306,307,308,387,422,423,424,388,389,390,223,425,426,427,391,208,209,
210,520,521,522,211,212,213,523,431,432,433,318,319,320,524,525,526,324,325,326,527,312,313,314,348,349,350,528,529,530,428,429,
430,544,545,546,315,316,317,531,532,321,322,323,533,534,547,548,549,535,345,346,347,536,537,538,539,327,328,329,330,331,332,540,
437,438,439,541,542,543,512,333,334,335,342,343,344,513,514,434,435,436,515,550,551,552,516,517,518,351,553,554,555,519,336,337,
338,648,649,650,339,340,341,559,560,561,651,446,447,448,652,653,654,452,453,454,655,440,441,442,476,477,478,656,657,658,556,557,
558,672,673,674,443,444,445,659,660,449,450,451,661,662,675,676,677,663,473,474,475,664,665,666,455,456,457,667,458,459,460,668,
565,566,567,669,670,671,640,461,462,463,470,471,472,641,642,562,563,564,643,644,678,679,680,645,646,479,464,465,466,647,681,682,
683,776,777,778,467,468,469,779,687,688,689,574,575,576,780,781,782,580,581,582,783,568,569,570,604,605,606,784,785,786,800,801,
802,571,572,573,787,684,685,686,788,577,578,579,789,790,803,804,805,791,601,602,603,792,793,794,795,583,584,585,586,587,588,796,
797,798,693,694,695,799,768,589,590,591,598,599,600,769,770,771,772,806,807,808,690,691,692,773,774,607,775,592,593,594,809,810,
811,904,905,906,595,596,597,907,815,816,817,908,909,910,702,703,704,911,912,913,696,697,698,708,709,710,914,928,929,930,812,813,
814,915,732,733,734,916,917,699,700,701,931,932,933,918,919,705,706,707,920,921,922,729,730,731,923,924,925,821,822,823,926,711,
712,713,927,714,715,716,896,897,898,934,935,936,899,818,819,820,900,717,718,719,726,727,728,901,937,938,939,902,903,735,720,721,
722,943,944,945,830,831,832,723,724,725,824,825,826,836,837,838,940,941,942,860,861,862,827,828,829,833,834,835,857,858,859,949,
950,951,839,840,841,842,843,844,946,947,948,845,846,847,854,855,856,863,848,849,850,958,959,960,988,989,990,964,965,966,851,852,
853,952,953,954,955,956,957,961,962,963,985,986,987,967,968,969,970,971,972,973,974,975,982,983,984,991,976,977,978,979,980,981,
};

struct RunTab { unsigned v[256]; };
constexpr RunTab make_runs() {
  RunTab t{};
  for (int b = 0; b < 256; ++b) { int k = 0; for (int i = 0; i < 8; ++i) { if ((b >> i) & 1) ++k; else t.v[b] += 1u << (4 * k); } }
  return t;
}
__device__ const __attribute__((aligned(16))) RunTab RUNS = make_runs();

struct Params {
  const float* x; const float* w;
  char* out; char* wsl;
  int rank, pad;
  char* ws[NR];
};

__device__ __forceinline__ unsigned ld_ag(const char* p) { return __hip_atomic_load((const GAS unsigned*)p, __ATOMIC_RELAXED, __HIP_MEMORY_SCOPE_AGENT); }
__device__ __forceinline__ void st_sys64(char* p, u64 v) { __hip_atomic_store((GAS u64*)p, v, __ATOMIC_RELAXED, __HIP_MEMORY_SCOPE_SYSTEM); }
__device__ __forceinline__ void st_ag(char* p, unsigned v) { __hip_atomic_store((GAS unsigned*)p, v, __ATOMIC_RELAXED, __HIP_MEMORY_SCOPE_AGENT); }
__device__ __forceinline__ void add_ag(char* p, unsigned v) { (void)__hip_atomic_fetch_add((GAS unsigned*)p, v, __ATOMIC_RELAXED, __HIP_MEMORY_SCOPE_AGENT); }
__device__ __forceinline__ int uni(int v) { return __builtin_amdgcn_readfirstlane(v); }
__device__ __forceinline__ __amdgpu_buffer_rsrc_t mk_rsrc(const void* p, unsigned bytes) { return __builtin_amdgcn_make_buffer_rsrc((void*)p, 0, (int)bytes, 0x00020000); }
__device__ __forceinline__ unsigned pk_bf16(float a, float b) { f32x2 v = {a, b}; return __builtin_bit_cast(unsigned, __builtin_convertvector(v, bf16x2)); }
#define WAIT_V(n) asm volatile("s_waitcnt vmcnt(" #n ")" ::: "memory")
#define WAIT_V0() do { WAIT_V(0); __builtin_amdgcn_s_waitcnt(0x0F70); } while (0)
#define FENCE() asm volatile("" ::: "memory")
#define BAR() do { asm volatile("" ::: "memory"); __builtin_amdgcn_s_waitcnt(0xC07F); __builtin_amdgcn_s_barrier(); asm volatile("" ::: "memory"); } while (0)
#define BFI(m, a, b) (((a) & (m)) | ((b) & ~(m)))
__device__ __forceinline__ unsigned GSEL(unsigned g) { unsigned r; asm("v_mad_u32_u24 %0, %1, %2, %3" : "=v"(r) : "v"(g), "s"(0x404040u), "v"(g)); return r; }
#define SCHED() __builtin_amdgcn_sched_barrier(0)
#define NOW() __builtin_amdgcn_s_memrealtime()
#define ST128(d, r, vo, so, aux) do { const u32x4 d_ = (d); __builtin_amdgcn_raw_buffer_store_b128(d_, r, vo, so, aux); asm volatile("s_nop 1" :: "v"(d_) : "memory"); } while (0)

__device__ __forceinline__ bool give_up(char* wsl, unsigned call, u64 t0) {
  const u64 el = NOW() - t0;
  if (el > 100000ull && (el > DEADLINE || ld_ag(wsl + OFF_TMO) == call)) { st_ag(wsl + OFF_TMO, call); return true; }
  __builtin_amdgcn_s_sleep(2);
  return false;
}

__device__ __forceinline__ unsigned scan_incl(unsigned v) {
  v += __builtin_amdgcn_update_dpp(0, (int)v, 0x111, 0xf, 0xf, false);
  v += __builtin_amdgcn_update_dpp(0, (int)v, 0x112, 0xf, 0xf, false);
  v += __builtin_amdgcn_update_dpp(0, (int)v, 0x114, 0xf, 0xf, false);
  v += __builtin_amdgcn_update_dpp(0, (int)v, 0x118, 0xf, 0xf, false);
  v += __builtin_amdgcn_update_dpp(0, (int)v, 0x142, 0xa, 0xf, false);
  v += __builtin_amdgcn_update_dpp(0, (int)v, 0x143, 0xc, 0xf, false);
  return v;
}
__device__ __forceinline__ int wave_max(int v) {
  v = max(v, __builtin_amdgcn_update_dpp(0, v, 0x111, 0xf, 0xf, false));
  v = max(v, __builtin_amdgcn_update_dpp(0, v, 0x112, 0xf, 0xf, false));
  v = max(v, __builtin_amdgcn_update_dpp(0, v, 0x114, 0xf, 0xf, false));
  v = max(v, __builtin_amdgcn_update_dpp(0, v, 0x118, 0xf, 0xf, false));
  v = max(v, __builtin_amdgcn_update_dpp(0, v, 0x142, 0xa, 0xf, false));
  v = max(v, __builtin_amdgcn_update_dpp(0, v, 0x143, 0xc, 0xf, false));
  return __builtin_amdgcn_readlane(v, 63);
}
__device__ __forceinline__ int max8(int v) {
  v = max(v, __builtin_amdgcn_update_dpp(0, v, 0x111, 0xf, 0xf, false));
  v = max(v, __builtin_amdgcn_update_dpp(0, v, 0x112, 0xf, 0xf, false));
  v = max(v, __builtin_amdgcn_update_dpp(0, v, 0x114, 0xf, 0xf, false));
  return __builtin_amdgcn_readlane(v, 7);
}
__device__ __forceinline__ unsigned piece_offsets(const u32x3 mk, unsigned (&off)[6]) {
  unsigned base = 0;
#pragma unroll
  for (int D = 0; D < 3; ++D) {
    const unsigned c = __builtin_popcount(mk[D] & 0x0f0f0f0fu) | (__builtin_popcount(mk[D] & 0xf0f0f0f0u) << 16);
    const unsigned s = scan_incl(c), ex = s - c, tot = (unsigned)__builtin_amdgcn_readlane((int)s, 63);
    off[2 * D] = base + (ex & 0xffffu); base += tot & 0xffffu;
    off[2 * D + 1] = base + (ex >> 16); base += tot >> 16;
  }
  return base;
}

__device__ __forceinline__ unsigned chunk_offsets(const unsigned (&k)[4], unsigned (&off)[4]) {
  unsigned base = 0;
#pragma unroll
  for (int D = 0; D < 2; ++D) {
    const unsigned c = k[2 * D] | (k[2 * D + 1] << 16), s = scan_incl(c), ex = s - c, tot = (unsigned)__builtin_amdgcn_readlane((int)s, 63);
    off[2 * D] = base + (ex & 0xffffu); base += tot & 0xffffu;
    off[2 * D + 1] = base + (ex >> 16); base += tot >> 16;
  }
  return base;
}

#define UNARY8(a, b, st, ln, big) { \
    unsigned p0 = (a) + ((a) << 8); p0 += p0 << 16; \
    unsigned p1 = (b) + (p0 >> 24); p1 += p1 << 8; p1 += p1 << 16; \
    const unsigned sp = ((1u << (p0 & 31u)) | (1u << ((p0 >> 8) & 31u)) | (1u << ((p0 >> 16) & 31u)) | (1u << ((p0 >> 24) & 31u)) | \
                         (1u << (p1 & 31u)) | (1u << ((p1 >> 8) & 31u)) | (1u << ((p1 >> 16) & 31u)) | (1u << ((p1 >> 24) & 31u))) >> 1; \
    st |= (u64)sp << (ln & 63u); ln += p1 >> 24; big |= p1; }
#define PUTBITS(base, pos, st, ln) if (ln) { \
    const unsigned ad = (base) + ((pos) >> 5) * 4u, sh = (pos) & 31u; \
    const u64 sv = (st) << sh; \
    asm volatile("ds_or_b32 %0, %1\n\tds_or_b32 %0, %2 offset:4" :: "v"(ad), "v"((unsigned)sv), "v"((unsigned)(sv >> 32)) : "memory"); \
    if (sh + (ln) > 64u) asm volatile("ds_or_b32 %0, %1 offset:8" :: "v"(ad), "v"((unsigned)((st) >> 32) >> (32u - sh)) : "memory"); }

#define RUNS_PUT() { asm volatile("" : "+v"(t[0]), "+v"(t[1]), "+v"(t[2]), "+v"(t[3])); \
            if (zo < ne + xb) { _Pragma("unroll") for (int z = 0; z < 4; ++z) { \
              const u64 sv = (u64)t[z] << ((4u * zo) & 31u); \
              asm volatile("ds_add_u32 %0, %1\n\tds_add_u32 %0, %2 offset:4" :: "v"((zo >> 1) & ~3u), "v"((unsigned)sv), "v"((unsigned)(sv >> 32)) : "memory"); \
              zo += __builtin_popcount(sw & (0xffu << (8 * z))); } } }
__device__ __forceinline__ void conv_load(const Params& P, int U, int wid, int lane, f32x4 (&v)[8]) {
  const int t = U / 80, r = U % 80, s = wid, c = lane & 15, kg = lane >> 4;
  const bool isB = r < 64;
  const int dest = (P.rank + 1 + (r & 7)) & 7, j = r >> 3, i = r - 64;
  __amdgpu_buffer_rsrc_t ri = mk_rsrc(isB ? P.w : P.x, U >= NUNIT ? 0u : isB ? KSH * GN * 4u : GM * KSH * 4u);
  const unsigned vo = isB ? (kg * 4 * GN + c * 4) * 4u : (c * KSH + kg * 4) * 4u;
  const unsigned so = isB ? ((t * 64 + (s >> 2) * 32) * GN + dest * NLOC + j * 256 + (s & 3) * 64) * 4u : ((i * 256 + (s & 3) * 64) * KSH + t * 64 + (s >> 2) * 32) * 4u;
  if (isB) {
#pragma unroll
    for (int q = 0; q < 8; ++q) v[q] = __builtin_bit_cast(f32x4, __builtin_amdgcn_raw_buffer_load_b128(ri, vo, so + ((q >> 2) * 16 + (q & 3)) * GN * 4u, AUX_NT));
  } else {
#pragma unroll
    for (int q = 0; q < 8; ++q) v[q] = __builtin_bit_cast(f32x4, __builtin_amdgcn_raw_buffer_load_b128(ri, vo, so + ((q >> 1) * 16 * KSH + (q & 1) * 16) * 4u, 0));
  }
  FENCE();
}
__device__ __forceinline__ void conv_unit(const Params& P, char* wsl, char*& pw, LAS char* lds, int U, int UN, f32x4 (&v)[8], int wid, int lane, unsigned prev, int hm) {
  const int rank = P.rank;
  const bool act = (hm >> (wid >> 2)) & 1;
  const int t = U / 80, r = U % 80, s = wid;
  const int kt = t * 8 + rank;
  const bool isB = r < 64;
  const int dest = (rank + 1 + (r & 7)) & 7, j = r >> 3, i = r - 64;
  __amdgpu_buffer_rsrc_t rl = mk_rsrc(wsl, WS_NEED);
  unsigned f[16];
  if (isB) {
#pragma unroll
    for (int n = 0; n < 4; ++n)
#pragma unroll
      for (int d = 0; d < 4; ++d) f[n * 4 + d] = pk_bf16(v[2 * d][n], v[2 * d + 1][n]);
  } else {
#pragma unroll
    for (int q = 0; q < 8; ++q) { f[2 * q] = pk_bf16(v[q][0], v[q][1]); f[2 * q + 1] = pk_bf16(v[q][2], v[q][3]); }
  }
  const bool local = !isB || dest == rank, remote = !isB || dest != rank;
  if (remote && act) {
    const bool push = U < 64;
    const unsigned e = (unsigned)((t * 80 + (isB ? (push ? rank : dest) * 8 + j : r)) * 8 + s);
    __amdgpu_buffer_rsrc_t rx = mk_rsrc(push ? P.ws[dest] : wsl, WS_NEED);
    LAS char* wb = lds + wid * 2048;
    *(LAS u32x4*)(wb + 512 + lane * 16) = (u32x4){0u, 0u, 0u, 0u};
    u32x4 M0, M1; unsigned E[8], sum = 0;
#pragma unroll
    for (int q = 0; q < 8; ++q) {
      const unsigned r0 = __builtin_amdgcn_alignbit(f[2 * q], f[2 * q], 31), r1 = __builtin_amdgcn_alignbit(f[2 * q + 1], f[2 * q + 1], 31);
      const unsigned m = __builtin_amdgcn_perm(r1, r0, 0x06040200u);
      if (q < 4) M0[q] = m; else M1[q - 4] = m;
      E[q] = __builtin_amdgcn_perm(r1, r0, 0x07050301u);
      sum = __builtin_amdgcn_sad_u8(E[q], 0u, sum);
    }
    ST128(M0, rx, lane * 16, OFF_XLE + e * XES, AUX_SYS);
    ST128(M1, rx, lane * 16 + 1024, OFF_XLE + e * XES, AUX_SYS);
    const unsigned wlo = (unsigned)min(max(((__builtin_amdgcn_readlane((int)scan_incl(sum), 63) + 4014) >> 11) - 14, 3), 240);
    const unsigned WL = wlo * 0x01010101u;
    unsigned W[8], L[8], any = 0; u32x2 C = {0u, 0u};
#pragma unroll
    for (int q = 0; q < 8; ++q) {
      W[q] = ((E[q] | 0x80808080u) - (WL & 0x7f7f7f7fu)) ^ ((E[q] ^ ~WL) & 0x80808080u);
      any |= W[q];
      const unsigned z = W[q] & 0x0f0f0f0fu;
      const unsigned y = __builtin_amdgcn_perm(0xf9f8f709u, 0xf5f4f3f2u, z) ^ __builtin_amdgcn_perm(0xfccfdf10u, 0xfefdfbfau, z ^ 0x08080808u);
      L[q] = y & 0x0f0f0f0fu;
      C[q >> 2] |= ((q & 3) < 2 ? y >> (4 - 2 * (q & 3)) : y << (2 * (q & 3) - 4)) & (0x03030303u << (2 * (q & 3)));
    }
    *(LAS u32x2*)(wb + lane * 8) = C;
    unsigned nexc = 0;
    if (__ballot((any & 0xf0f0f0f0u) != 0u)) {
      LAS unsigned* hd = (LAS unsigned*)(wb + 512);
#pragma unroll
      for (int q = 0; q < 8; ++q) {
        unsigned ov = (((W[q] & 0x70707070u) + 0x70707070u) | W[q]) & 0x80808080u;
        for (;;) {
          const u64 bal = __ballot(ov != 0u);
          if (!bal || nexc > 15u) break;
          const unsigned sl = nexc + __builtin_amdgcn_mbcnt_hi((unsigned)(bal >> 32), __builtin_amdgcn_mbcnt_lo((unsigned)bal, 0u));
          if (ov != 0u && sl < 15u) { const unsigned sh = __builtin_ctz(ov) & 24u; hd[1 + sl] = (unsigned)lane | (q << 8) | (sh << 16) | (((E[q] >> sh) & 0xffu) << 24); }
          ov &= ov - 1;
          nexc += __builtin_popcountll(bal);
        }
      }
      if (lane == 0) hd[0] = nexc;
    }
    u64 st = 0; unsigned ln = 0, big = 0;
#pragma unroll
    for (int h = 0; h < 4; ++h) UNARY8(L[2 * h], L[2 * h + 1], st, ln, big)
    big |= ln + 0x1fffffbfu;
    const unsigned cn = ln | (__builtin_popcountll(st) << 16), sc = scan_incl(cn), tot = (unsigned)__builtin_amdgcn_readlane((int)sc, 63);
    const unsigned nh = nexc ? nexc + 1u : 0u, nd = ((tot & 0xffffu) + 31u) >> 5, nt = nh + nd;
    unsigned mode = (wlo - 3u) | ((nt + 31u) >> 5 << 8) | (nexc ? 0x800u : 0u);
    if (nexc > 15u || nd > 64u || (tot >> 16) > 1016u || __ballot(big >= 0x20000000u)) {
      mode = 0x700u;
      const u32x4 E0 = {E[0], E[1], E[2], E[3]}, E1 = {E[4], E[5], E[6], E[7]};
      ST128(E0, rx, lane * 16 + 2048, OFF_XLE + e * XES, AUX_SYS);
      ST128(E1, rx, lane * 16 + 3072, OFF_XLE + e * XES, AUX_SYS);
    } else {
      PUTBITS((unsigned)(size_t)wb + 512u + nh * 4u, (sc - cn) & 0xffffu, st, ln)
      FENCE();
      if (lane * 4u < 128u + nt) ST128(*(const LAS u32x4*)(wb + lane * 16), rx, lane * 16 + 2048, OFF_XLE + e * XES, AUX_SYS);
      if (U >= 80 && nt > 32u && nt <= 40u) mode = (mode & ~0x700u) | 0x500u;
    }
    if (lane == 0) *(volatile LAS unsigned short*)(lds + LDS_CTL + 128 + wid * 2) = (unsigned short)mode;
  }
  if (local && act) {
    const unsigned base = isB ? OFF_STB + (unsigned)(((kt * 2 + (s >> 2)) * 8 + j) * 4 + (s & 3)) * 4096u : OFF_STA + (unsigned)((kt * 2 + (s >> 2)) * 64 + i * 4 + (s & 3)) * 4096u;
#pragma unroll
    for (int n = 0; n < 4; ++n) { const u32x4 o = {f[4 * n], f[4 * n + 1], f[4 * n + 2], f[4 * n + 3]}; ST128(o, rl, lane * 16 + n * 1024, base, AUX_AG); }
  }
  if (remote && U >= 80) {
    BAR();
    if (wid != rank && (!isB || wid == dest)) {
      const u32x4 d = *(const LAS u32x4*)(lds + ((lane >> 1) & 7) * 2048 + 640 + (lane & 1) * 16);
      const unsigned oo = OFF_OVF + rank * 49152 + (t * 24 + (isB ? j : 8 + i)) * 256;
      ST128(d, mk_rsrc(P.ws[wid], oo + 256u), lane * 16, oo, AUX_SYS);
    }
  }
  if (!act) __builtin_amdgcn_s_sleep(50);
  conv_load(P, UN, wid, lane, v);
  if (UN < NUNIT) WAIT_V(8); else WAIT_V(0);
  asm volatile("" : "+v"(pw));
  if (local && act && lane == 0) add_ag(wsl + (isB ? OFF_CNTB + j * CSTR : OFF_CNTA + ((i * 4 + (s & 3)) / 3) * CSTR) + kt * 4, 1u);
  BAR();
  if (wid == 0) {
    if (remote && lane < 2 * NR) {
      const int pd = lane >> 1, hf = lane & 1;
      const unsigned p0 = *(volatile LAS unsigned*)(lds + LDS_CTL + 128 + hf * 8), p1 = *(volatile LAS unsigned*)(lds + LDS_CTL + 132 + hf * 8);
      const u64 pay = (u64)((p0 & 0xfffu) | (p0 >> 16 << 12)) | ((u64)((p1 & 0xfffu) | (p1 >> 16 << 12)) << 24);
      if (pd != rank && (!isB || pd == dest) && ((hm >> hf) & 1))
        st_sys64(pw + OFF_UFLAG + rank * CSTR + (t * 24 + (isB ? j : 8 + i)) * 16 + hf * 8, (pay << 16) | ((prev + 1u) & 0xffffu));
    }
  }
  BAR();
}

__global__ __launch_bounds__(512, 2) void fused_kernel(Params P) {
  __shared__ __attribute__((aligned(1024))) char smem[LDS_BYTES];
  const int tid = threadIdx.x, lane = tid & 63, wid = uni(tid >> 6);
  const int b = blockIdx.x, rank = P.rank;
  char* wsl = P.wsl;
  LAS char* lds = (LAS char*)smem;
  const unsigned prev = ld_ag(wsl + OFF_CALL + b * 64);
  char* pw = P.ws[(lane >> 1) & 7];

  bool bad = false;
  int dw = b < GEMM0 - 1 && (7 - wid) * (GEMM0 - 1) + b < 7 * DPL ? (7 - wid) * (GEMM0 - 1) + b : 1 << 20;
  if (b < GEMM0) {
    if (b < NLINKWG) {
      const int rel = b % 7, src = (rank + 1 + rel) & 7;
      {
        int ens[3];
#pragma unroll
        for (int q = 0; q < 3; ++q) ens[q] = ORD[rel * 768 + q * 256 + (tid >> 1)];
        const unsigned rv = RUNS.v[tid & 255];
        if (tid < TASKS_ALL - TASKS_IN) {
          const unsigned mt = tid + 16 * PMP;
          const u32x4 d = {OFF_EXM + mt * 3072u, OFF_LM + ((unsigned)src * (NSUB / 4) + mt) * 3072u, OFF_MRC + src * CSTR + (mt >> 4) * 4, ((mt >> 4) * 8 + (mt & 7)) * 16 + 0x8000u + (16 << 16)};
          *(LAS u32x4*)(lds + 106496 + (TASKS_IN + tid) * 16) = d;
        }
#pragma unroll
        for (int q = 0; q < 3; ++q) {
          const int i = q * 512 + tid, en = ens[q], t = en >> 7, u = (en & 127) >> 2, s = (i & 1) * 4 + (en & 3), kt = t * 8 + src, grp = (u - 8) * 4 + (s & 3);
          u32x4 d;
          d[0] = OFF_XLE + (unsigned)((t * 80 + (u < 8 ? (i < NPUSH ? src : rank) * 8 + u : 56 + u)) * 8 + s) * XES;
          d[1] = u < 8 ? OFF_STB + (unsigned)(((kt * 2 + (s >> 2)) * 8 + u) * 4 + (s & 3)) * 4096u : OFF_STA + (unsigned)((kt * 2 + (s >> 2)) * 64 + grp) * 4096u;
          d[2] = (u < 8 ? OFF_CNTB + u * CSTR : OFF_CNTA + (grp / 3) * CSTR) + kt * 4;
          d[3] = (t * 24 + u) * 16 + (s >> 2) * 8 + ((16 + 12 * (s & 3)) << 16) + ((s & 3) << 30);
          *(LAS u32x4*)(lds + 106496 + i * 16) = d;
        }
        *(LAS unsigned*)(lds + 59392 + (tid & 255) * 4) = rv;
      }
      BAR();
      if (wid != PWV || b < 7 * XW) {
        const unsigned call = (unsigned)uni((int)prev) + 1u;
        constexpr int L0 = PWV + 1;
        const bool loc = wid >= L0;
        const int k = loc ? (b / 7) * (8 - L0) + wid - L0 : wid == PWV ? 8 * PWV + b / 7 : (b / 7) * PWV + wid;
        __amdgpu_buffer_rsrc_t rl = mk_rsrc(wsl, WS_NEED);
        const unsigned tbase = (call - 1u) * (unsigned)(TASKS_ALL - NPUSH) - (unsigned)(2 * WPL + NPUSH), nend = loc ? NPUSH : TASKS_ALL;
        char* tp = wsl + OFF_TICK + rel * CSTR;
        const u32x4 z4 = {call - 1u, 0, call - 1u, 0};
        u32x4 vv[2][4] = {{z4, z4, z4, z4}, {z4, z4, z4, z4}}; unsigned n = (unsigned)(loc ? k : k + NPUSH), n2 = n + (loc ? 64u - 8 * L0 : WPL);
        LAS char* wb = lds + 60416 + wid * 2688;
        LAS char* fc = lds + 81920 + wid * 3072;
#pragma unroll
        for (int q = 0; q < 3; ++q) *(LAS u32x4*)(fc + q * 1024 + lane * 16) = z4;
        char* wsr = loc ? wsl : P.ws[src];
        unsigned cur_d = 0, cur_c = 0, cur_m = 0, prv_c = 0, nxt_d = 0, nxt_c = 0, nxt_m = 0, so = 0, lim = 0, s3 = 0, l3 = 0;
        char* b3 = wsl;
        bool cur = false, prv = false, nxt = false;
        bool once = loc;
        const u64 t0 = NOW();
        do {
#pragma unroll
        for (int ph = 0; ph < 2; ++ph) {
          u32x4 (&v)[4] = vv[ph];
          const u32x4 (&o)[4] = vv[ph ^ 1];
          {
            __amdgpu_buffer_rsrc_t rs = mk_rsrc(wsr, lim);
#pragma unroll
            for (int q = 0; q < 3; ++q) v[q] = __builtin_amdgcn_raw_buffer_load_b128(rs, lane * 16 + q * 1024, so, AUX_SYS);
            v[3] = __builtin_amdgcn_raw_buffer_load_b128(mk_rsrc(b3, l3), lane * 16, s3, AUX_SYS);
          }
          FENCE();
          unsigned tk = (unsigned)__builtin_amdgcn_raw_ptr_buffer_atomic_add_i32(1, mk_rsrc(tp, nxt && !loc ? 4u : 0u), lane * 4, 0, AUX_AG);
          FENCE();
          if (nxt) n = n2;
          *(LAS u32x4*)(wb + lane * 16) = o[2];
          *(LAS u32x4*)(wb + ((cur_m & 0x700u) == 0x500u && lane < 4 ? 640 : 1536) + lane * 16) = o[3];
          *(LAS u32x2*)(wb + 1024 + lane * 8) = (u32x2){0u, 0u};
          FENCE();
          const u32x2 C = *(const LAS u32x2*)(wb + lane * 8);
          unsigned sw = *(const LAS unsigned*)(wb + 512 + lane * 4);
          const u32x4 d = *(const LAS u32x4*)(lds + 106496 + min(n, nend - 1u) * 16);
          FENCE();
          const unsigned ho = sw;
          if (cur_m & 0x800u) sw = *(const LAS unsigned*)(wb + 516 + ((uni((int)ho) & 15) + lane) * 4);
          unsigned t[4];
#pragma unroll
          for (int z = 0; z < 4; ++z) t[z] = *(const LAS unsigned*)(lds + 59392 + ((sw >> (8 * z)) & 0xffu) * 4);
          u32x2 f = *(const LAS u32x2*)(fc + (d[3] & 0xfffu));
          FENCE();
          const unsigned xb = ((unsigned)(size_t)wb + 1024u) * 2u;
          const unsigned F0 = ~(C[0] | (C[0] >> 1)) & 0x55555555u, F1 = ~(C[1] | (C[1] >> 1)) & 0x55555555u;
          const unsigned cn = __builtin_popcount(F0) + __builtin_popcount(F1) + (__builtin_popcount(sw) << 16), sc = scan_incl(cn);
          const unsigned ne = cur && (cur_m & 0x600u) != 0x600u ? (unsigned)__builtin_amdgcn_readlane((int)sc, 63) & 0xffffu : 0u;
          unsigned zo = ((sc - cn) >> 16) + xb, off = ((sc - cn) & 0xffffu) + xb;
          asm volatile("" : "+v"(f));
          RUNS_PUT()
          FENCE();
          unsigned of[4]; u32x2 sv[4];
#pragma unroll
          for (int h = 0; h < 4; ++h) { of[h] = off; off += __builtin_popcount((h < 2 ? F0 : F1) & (0x05050505u << (4 * (h & 1)))); }
#pragma unroll
          for (int h = 0; h < 4; ++h) sv[h] = *(const LAS u32x2a4*)(size_t)((of[h] >> 1) & ~3u);
          FENCE(); SCHED();
#define FLAG_OK() ((((unsigned)uni((int)f[0]) ^ call ^ (d3 & 0x8000u)) & 0xffffu) == 0u)
#define PREP() { nm = (unsigned)((((u64)(unsigned)uni((int)f[1]) << 32) | (unsigned)uni((int)f[0])) >> ((d3 >> 16) & 63u)) & 0xfffu;         \
            lim = ok ? so + 2560u + ((0xc4143210u >> ((nm >> 6) & 28u)) & 15u) * 128u : 0u; \
            const bool ovf = ok && (nm & 0x700u) == 0x500u; \
            b3 = ovf ? wsl : wsr; \
            s3 = ovf ? OFF_OVF + src * 49152u + ((d3 & 0xfffu) << 4) + (d3 >> 30 << 5) : so + 3072u; \
            l3 = ovf ? s3 + 32u : lim; }
          const unsigned d3 = (unsigned)uni((int)d[3]), nd = (unsigned)uni((int)d[1]), nc = (unsigned)uni((int)d[2]);
          so = (unsigned)uni((int)d[0]);
          bool ok = n < nend && FLAG_OK();
          unsigned nm;
          PREP()
          asm volatile("" : "+s"(lim), "+s"(s3), "+s"(l3), "+s"(nm));
          SCHED();
          WAIT_V(5); __builtin_amdgcn_s_waitcnt(0x0F75);
          (void)__builtin_amdgcn_raw_ptr_buffer_atomic_add_i32(1, mk_rsrc(wsl + prv_c, prv ? 4u : 0u), lane * 4, 0, 0);
          FENCE();
          u32x4 w[4] = {o[0], o[1], o[2], o[3]};
          if (cur && cur_m != 0x600u) {
            unsigned E[8];
            if ((cur_m & 0x700u) == 0x700u) {
#pragma unroll
              for (int q = 0; q < 4; ++q) { E[q] = o[2][q]; E[q + 4] = o[3][q]; }
            } else {
              const unsigned WL = (cur_m & 0xffu) * 0x01010101u;
              unsigned st[4];
#pragma unroll
              for (int h = 0; h < 4; ++h) st[h] = __builtin_amdgcn_alignbit(sv[h][1], sv[h][0], of[h] * 4u);
#pragma unroll
              for (int q = 0; q < 8; ++q) {
                const int sh = 2 * (q & 3);
                const unsigned F = q < 4 ? F0 : F1, g = (sh >= 2 ? F >> (sh - 2) : F << 2) & 0x04040404u;
                unsigned y = __builtin_amdgcn_perm(0u, st[q >> 1], 0x0c010c00u);
                y = (y | (y << 4)) & 0x0f0f0f0fu;
                y ^= 0x0f0f0f0fu ^ __builtin_amdgcn_perm(0u, 0x001f0301u, y);
                if (!(q & 1)) st[q >> 1] >>= 4u * __builtin_popcount(g);
                E[q] = __builtin_amdgcn_perm(y, 0x0e0e0e0eu, GSEL(g)) + ((C[q >> 2] >> sh) & 0x03030303u) + WL;
              }
              if (cur_m & 0x800u) {
                const int nx = uni((int)ho) & 15;
                for (int x = 1; x <= nx; ++x) {
                  const unsigned en = (unsigned)__builtin_amdgcn_readlane((int)ho, x);
                  const unsigned sh = (en >> 16) & 24u, val = (en >> 24) << sh, q0 = (en >> 8) & 7u;
                  const unsigned mk = (unsigned)lane == (en & 63u) ? 0xffu << sh : 0u;
#pragma unroll
                  for (int q = 0; q < 8; ++q) { const unsigned mq = q0 == (unsigned)q ? mk : 0u; E[q] = (E[q] & ~mq) | (val & mq); }
                }
              }
              FENCE();
            }
#pragma unroll
            for (int n = 0; n < 4; ++n) {
#pragma unroll
              for (int z = 0; z < 2; ++z) {
                const unsigned m = o[n >> 1][(n & 1) * 2 + z], ee = E[2 * n + z];
                const unsigned r0 = __builtin_amdgcn_perm(ee, m, 0x05010400u), r1 = __builtin_amdgcn_perm(ee, m, 0x07030602u);
                w[n][2 * z] = __builtin_amdgcn_alignbit(r0, r0, 1); w[n][2 * z + 1] = __builtin_amdgcn_alignbit(r1, r1, 1);
              }
            }
          }
          {
            __amdgpu_buffer_rsrc_t rw = mk_rsrc(wsl, cur ? cur_d + (cur_m == 0x600u ? 3072u : 4096u) : 0u);
#pragma unroll
            for (int q = 0; q < 4; ++q) ST128(w[q], rw, lane * 16 + q * 1024, cur_d, AUX_AG);
          }
          if (loc && cur) { WAIT_V0(); (void)__builtin_amdgcn_raw_ptr_buffer_atomic_add_i32(1, mk_rsrc(wsl + cur_c, 4u), lane * 4, 0, 0); FENCE(); cur = false; }
          prv = cur; prv_c = cur_c;
          cur = nxt; cur_d = nxt_d; cur_c = nxt_c; cur_m = nxt_m;
          if (n < nend && !ok) {
            const unsigned fo = (d3 & 0x8000u ? OFF_MFLAG : OFF_UFLAG) + src * CSTR, g = d3 & 0xc00u;
            do {
              __builtin_amdgcn_raw_ptr_buffer_load_lds(rl, fc + g, 16, lane * 16, fo + g, 0, AUX_SYS);
              if (g < 0x800u && (d3 & 0x3ffu) >= 24 * 16) __builtin_amdgcn_raw_ptr_buffer_load_lds(rl, fc + g, 16, lane * 16, fo + g, 1024, AUX_SYS);
              WAIT_V0();
              f = *(const LAS u32x2*)(fc + (d3 & 0xfffu));
              ok = FLAG_OK() || bad;
              if (!ok && !cur && !prv && give_up(wsl, call, t0)) bad = true;
              if (once && !ok) { __builtin_amdgcn_s_sleep(52); __builtin_amdgcn_s_sleep(52); }
              once = false;
              WAIT_V0();
            } while (!ok && !cur && !prv && !(d3 & 0x8000u));
            PREP()
          }
          nxt = ok; nxt_d = nd; nxt_c = nc; nxt_m = nm;
          SCHED();
          WAIT_V(5); __builtin_amdgcn_s_waitcnt(0x0F75);
          asm volatile("" : "+v"(v[0]), "+v"(v[1]), "+v"(v[2]), "+v"(v[3]), "+v"(tk));
          if (cur) n2 = loc ? n + (64u - 8 * L0) : (unsigned)uni((int)tk) - tbase;
        }
        } while (n < nend || cur || prv);
      }
    } else {
      const int cw = b - NLINKWG;
      int U = cw < 16 ? 64 + cw : 2 * NGEMM + cw, hm = cw < 16 ? 2 : 3;
      if (hm == 3) __builtin_amdgcn_s_sleep(50);
      f32x4 v[8];
      conv_load(P, hm == 2 && wid < 4 ? NUNIT : U, wid, lane, v);
      while (U < NUNIT) {
        const int UN = hm == 2 ? 2 * NGEMM + cw : U + NCONVWG;
        conv_unit(P, wsl, pw, lds, U, UN, v, wid, lane, prev, hm);
        U = UN; hm = 3;
      }
    }
    if (b == GEMM0 - 1) {
      const unsigned call = (unsigned)uni((int)prev) + 1u;
      __amdgpu_buffer_rsrc_t rm = mk_rsrc(wsl + OFF_AMX + wid * CSTR, NGEMM * 8);
      const u64 t0 = NOW();
      u32x4 g0, g1;
      for (;;) {
        g0 = __builtin_amdgcn_raw_buffer_load_b128(rm, lane * 16, 0, AUX_SYS);
        g1 = __builtin_amdgcn_raw_buffer_load_b128(rm, (lane < NGEMM / 2 - 64 ? lane + 64 : lane) * 16, 0, AUX_SYS); FENCE();
        const u64 ok = __ballot(g0[1] == call && g0[3] == call && g1[1] == call && g1[3] == call);
        if (ok == ~0ull) break;
        if (!__any(g0[1] == call)) __builtin_amdgcn_s_sleep(30);
        if (give_up(wsl, call, t0)) break;
      }
      const int mx = wave_max(max(max((int)g0[0], (int)g0[2]), max((int)g1[0], (int)g1[2])));
      volatile LAS int* red = (volatile LAS int*)(lds + LDS_CTL + 64);
      if (lane == 0) red[wid] = mx;
      BAR();
      if (wid == 0) {
        const unsigned am = (unsigned)max8(red[lane & 7]);
        if (lane < 16) __hip_atomic_store((GAS u64*)(wsl + OFF_AMF + lane * 256), ((u64)call << 32) | am, __ATOMIC_RELAXED, __HIP_MEMORY_SCOPE_AGENT);
      }
    }
  } else {
    const int g = b - GEMM0;
    if (g >= 80) __builtin_amdgcn_s_sleep(50);
    {
      f32x4 v[8];
      const int hm = (g >> 4) == 4 ? 1 : 3;
      conv_load(P, hm == 1 && wid >= 4 ? NUNIT : g, wid, lane, v);
      conv_unit(P, wsl, pw, lds, g, NUNIT, v, wid, lane, prev, hm);
      int U2 = g;
      {
        const int x = g & 7, i = g >> 3, e = (x >> 1) * 4 + i, ie = i < 6 ? i : i < 16 ? i - 2 : i - 4;
        if (!(x & 1) && i < 4) U2 = (e < 6 ? e : e < 14 ? e + 2 : e + 4) * 8 + 7;
        if (x == 7 && (i < 6 || (i >= 8 && i < 16) || i == 18 || i == 19)) U2 = (ie & 3) * 8 + (ie >> 2) * 2;
      }
      conv_load(P, NGEMM + U2, wid, lane, v);
      conv_unit(P, wsl, pw, lds, NGEMM + U2, NUNIT, v, wid, lane, prev, 3);
      WAIT_V0();
    }
    const unsigned call = (unsigned)uni((int)prev) + 1u;
    const int L = (g & 7) * 22 + (g >> 3);
    const int pl = (L % 88) >> 2, pm = ((2 * pl) % 11) * 2 + (pl >= 11), pn = (L / 88) * 4 + (L & 3);
    const int wr = wid >> 2, wc = wid & 3;
    volatile LAS int* ctl = (volatile LAS int*)(lds + LDS_CTL);
    __amdgpu_buffer_rsrc_t rl = mk_rsrc(wsl, WS_NEED);
    const unsigned wantA = call * (pm == 21 ? 2u : 6u), wantB = call * 8u;
    f32x4 acc[6][4];
#pragma unroll
    for (int m = 0; m < 6; ++m)
#pragma unroll
      for (int n = 0; n < 4; ++n) acc[m][n] = (f32x4){0.f, 0.f, 0.f, 0.f};
    const unsigned gvoff = lane * 16;
    const bool real = wid >= 3 || wid == 0 || pm != 21;
    const unsigned gbase = !real ? WS_NEED : wid < 3 ? OFF_STA + pm * 12288u + wid * 4096u : OFF_STB + pn * 16384u + (wid - 3) * 4096u;
    const unsigned gsh = wid < 3 ? 18u : 17u;
    static_assert(NS == 3 && TASKS_IN == 3 * 512 && TASKS_ALL - TASKS_IN <= 512, "");
    const int gdst = wid * 4096;
    const int aoff = wr * 6144 + lane * 16, boff = 12288 + wc * 4096 + lane * 16;
    const unsigned pcopy = (unsigned)(size_t)(lds + LDS_CTL + 1536);
#define GL1(i) __builtin_amdgcn_raw_ptr_buffer_load_lds(rl, lds + slo + gdst, 16, so, gbase, (i) * 1024, 0)
#define GETE() { so = gvoff + ((unsigned)ev << gsh); }
#define NEXT() { slo = nxo; nxo = nxo == (NS - 1) * SLOT ? 0 : nxo + SLOT; ra = aoff + nxo; rbb = boff + nxo; asm volatile("" : "+v"(ra), "+v"(rbb)); }
#define MARK() { rm |= ((unsigned)uni(ev) >> 7) ^ 8u; asm volatile("" : "+s"(rm)); }
#define PC(P, i) __builtin_popcount((P) & ((1 << (i)) - 1))
#define RD1(NA, NB, r) do { if ((r) < 4) NB[(r) & 3] = *(const LAS bf16x8*)(lds + rbb + (r) * 1024); else NA[(r) < 4 ? 0 : (r) - 4] = *(const LAS bf16x8*)(lds + ra + ((r) - 4) * 1024); } while (0)
#define OP(P, i, NA, NB) if (((P) >> (i)) & 1) { SCHED(); if (PC(P, i) == 0) { GETE() SCHED(); } GL1(PC(P, i) & 3); SCHED(); } else RD1(NA, NB, (i) - PC(P, i));
#define RGP(P, NA, NB) do { unsigned so; \
      OP(P, 0, NA, NB) OP(P, 1, NA, NB) OP(P, 2, NA, NB) OP(P, 3, NA, NB) OP(P, 4, NA, NB) OP(P, 5, NA, NB) OP(P, 6, NA, NB) OP(P, 7, NA, NB) \
      OP(P, 8, NA, NB) OP(P, 9, NA, NB) OP(P, 10, NA, NB) OP(P, 11, NA, NB) OP(P, 12, NA, NB) OP(P, 13, NA, NB) SCHED(); } while (0)
#define POLL() do { __builtin_amdgcn_raw_ptr_buffer_load_lds(rl, lds + LDS_CTL + 1536, 4, lane * 4, OFF_CNTA + pm * CSTR, 0, AUX_AG); \
        if (needB) __builtin_amdgcn_raw_ptr_buffer_load_lds(rl, lds + LDS_CTL + 1792, 4, lane * 4, OFF_CNTB + pn * CSTR, 0, AUX_AG); } while (0)
#define RDALL(NA, NB) do { \
      _Pragma("unroll") for (int n = 0; n < 4; ++n) NB[n] = *(const LAS bf16x8*)(lds + rbb + n * 1024); \
      _Pragma("unroll") for (int m = 0; m < 6; ++m) NA[m] = *(const LAS bf16x8*)(lds + ra + m * 1024); SCHED(); } while (0)
#define MM(CA, CB) do { if (rm & 1u) { \
      _Pragma("unroll") for (int m = 0; m < 6; ++m) \
        _Pragma("unroll") for (int n = 0; n < 4; ++n) acc[m][n] = __builtin_amdgcn_mfma_f32_16x16x32_bf16(CB[n], CA[m], acc[m][n], 0, 0, 0); \
      ++ndone; } SCHED(); } while (0)
#define STEP(G, par, CA, CB, NA, NB) { \
      if (G == 0) { const int ev = ctl[(par) ^ 1]; MM(CA, CB); RGP(PT0, NA, NB); MARK() NEXT() SCHED(); WAIT_V(4); } \
      else if (G == 1) { const int ev = ctl[(par) ^ 1]; SCHED(); RGP(PT1, NA, NB); MARK() NEXT() SCHED(); MM(CA, CB); WAIT_V(4); } \
      else { \
        rm |= last >= 0 ? 1u << NS : 0u; \
        const bool dec = (last & 1) != 0;                   \
        if (dec && left == 0ull && ready != ~0ull) {        \
          if (rm == 0u) {                                   \
            if (ndone > 0 && ndone < FAST_AT) __builtin_amdgcn_s_sleep(40); \
            POLL(); WAIT_V0(); cost = 0; \
            if ((++spins & 63) == 0 && give_up(wsl, call, t0)) { left = ~ready; ready = ~0ull; } \
          } \
          unsigned pa, pb; \
          asm volatile("ds_read_b32 %0, %2\n\tds_read_b32 %1, %2 offset:256" : "=&v"(pa), "=&v"(pb) : "v"(pcopy + lane * 4) : "memory"); \
          SCHED(); RDALL(NA, NB); \
          asm volatile("s_waitcnt lgkmcnt(10)" : "+v"(pa), "+v"(pb) :: "memory"); \
          const u64 bA = __builtin_amdgcn_ballot_w64(pa >= wantA), bB = __builtin_amdgcn_ballot_w64(pb >= wantB); \
          left |= bA & bB & ~ready; \
          ready |= left; \
          needB = (bA & ~bB) != 0ull || (bB != ~0ull && __builtin_popcountll(bB & ~ready) < BTH);     \
        } else RDALL(NA, NB); \
        last = dec ? (left ? 2 * __builtin_ctzll(left) : -1) : last + 1; \
        if (dec) left &= left - 1; \
        ctl[par] = last & 0x47f; \
        if (ready != ~0ull) { \
          cost += (rm & 1u) ? 3 : 1; \
          if ((left & (left - 1)) == 0ull && cost >= (ndone < FAST_AT ? POLL_SLOW : POLL_COST)) { cost = 0; WAIT_V(12); POLL(); } \
        } \
        NEXT() SCHED(); MM(CA, CB); } \
      BAR(); rm >>= 1;  }
    bf16x8 xa[6], xb[4], ya[6], yb[4];
#pragma unroll
    for (int m = 0; m < 6; ++m) { xa[m] = (bf16x8)(__bf16)0.f; ya[m] = xa[m]; }
#pragma unroll
    for (int n = 0; n < 4; ++n) { xb[n] = (bf16x8)(__bf16)0.f; yb[n] = xb[n]; }
    int slo = 0, nxo = SLOT, ra = aoff + SLOT, rbb = boff + SLOT, ndone = 0; unsigned rm = 0;
    if (tid == 0) { ctl[1] = 0x47f; ctl[16] = 0; ctl[17] = 0; }
    if (tid < 128) ctl[384 + tid] = 0;
    const u64 t0 = NOW();
    BAR();
    {
      u64 left = 0, ready = 0; int last = -1, cost = 0; unsigned spins = 0; bool needB = true;
      if (wid < 4) { do { STEP(0, 0, xa, xb, xa, xb) STEP(0, 1, xa, xb, xa, xb)  } while (ndone != 128); }
      else if (wid < 7) { do { STEP(1, 0, xa, xb, ya, yb) STEP(1, 1, ya, yb, xa, xb) } while (ndone != 128); }
      else { do { STEP(2, 0, xa, xb, ya, yb) STEP(2, 1, ya, yb, xa, xb) } while (ndone != 128); }
    }
    if (pm == 21) {
#pragma unroll
      for (int m = 0; m < 6; ++m)
        if (wr == 1 || m >= 4) {
#pragma unroll
          for (int n = 0; n < 4; ++n) acc[m][n] = (f32x4){-1.f, -1.f, -1.f, -1.f};
        }
    }
    const int tile = pm * 8 + pn, sub = (pm * 2 + wr) * 32 + pn * 4 + wc;
    int mx = 0, my = 0;
#pragma unroll
    for (int m = 0; m < 6; ++m)
#pragma unroll
      for (int n = 0; n < 4; ++n)
        { mx = max(max(mx, __float_as_int(acc[m][n][0])), __float_as_int(acc[m][n][1])); my = max(max(my, __float_as_int(acc[m][n][2])), __float_as_int(acc[m][n][3])); }
    mx = wave_max(max(mx, my));
    volatile LAS int* red = (volatile LAS int*)(lds + LDS_CTL + 64);
    {
      unsigned cn = 0, tm = 0;
      WAIT_V0();
      if (lane == 0) asm volatile("ds_max_u32 %2, %3\n\tds_add_rtn_u32 %0, %2, %4 offset:4\n\tds_read_b32 %1, %2\n\ts_waitcnt lgkmcnt(0)" : "=&v"(cn), "=&v"(tm) : "v"((unsigned)(size_t)red), "v"(mx), "v"(1u) : "memory");
      mx = uni((int)tm);
      if (uni((int)cn) == 7 && lane < 2 * NR && !(lane & 1)) st_sys64(pw + OFF_AMX + rank * CSTR + tile * 8, ((u64)call << 32) | (unsigned)mx);
      FENCE();
    }
    const float thr = __int_as_float(mx) * (1.f / 255.f);
    if (wid * NGEMM + tile < 7 * DPG) dw = 7 * DPL + wid * NGEMM + tile;
    u32x3 mk;
#pragma unroll
    for (int D = 0; D < 3; ++D) {
      unsigned mm = 0;
#pragma unroll
      for (int n = 3; n >= 0; --n)
#pragma unroll
        for (int kk = 6; kk >= 0; kk -= 2) {
          const f32x2 d = (f32x2){thr, thr} - (f32x2){acc[2 * D + (kk >> 2)][n][kk & 3], acc[2 * D + (kk >> 2)][n][(kk & 3) + 1]};
          mm = __builtin_amdgcn_alignbit(__builtin_amdgcn_alignbit(mm, __float_as_uint(d[1]), 31), __float_as_uint(d[0]), 31);
        }
      mk[D] = mm;
    }
    *(LAS u32x3a4*)(lds + NS * SLOT + wid * 1024 + lane * 12) = mk;
    FENCE();
    const u32x4 mq = *(const LAS u32x4*)(lds + NS * SLOT + wid * 1024 + lane * 16);
    FENCE();
    if (lane < 48) {
      if (pm < PMP) {
#pragma unroll
        for (int p = 1; p < NR; ++p) {
          const int pd = ((rank + p + wid) & 7) == rank ? (rank + wid) & 7 : (rank + p + wid) & 7;
          const u64 base = ((u64)(unsigned)__builtin_amdgcn_readlane((int)((u64)pw >> 32), 2 * pd) << 32) | (unsigned)__builtin_amdgcn_readlane((int)(u64)pw, 2 * pd);
          ST128(mq, mk_rsrc((const void*)base, WS_NEED), lane * 16, OFF_LM + (unsigned)(rank * NSUB + sub) * 768u, AUX_SYS);
        }
      } else ST128(mq, rl, lane * 16, OFF_EXM + (unsigned)sub * 768u, AUX_SYS);
    }
    if (pm >= PMP) WAIT_V(0);
    BAR();
    if (pm >= PMP && wid == 1 && lane < 2 * NR && !(lane & 1) && (lane >> 1) != rank) st_sys64(pw + OFF_MFLAG + rank * CSTR + tile * 16, 0x6000000u | ((call ^ 0x8000u) & 0xffffu));
    unsigned off[6];
    const unsigned cnt = piece_offsets(mk, off), cnt1 = (unsigned)uni((int)off[4]), ng = (cnt + 31u) >> 5;
    f32x4 cf[3][8];
    LAS char* lw = lds + wid * 18432;
    {
      const unsigned lb = wid * 18432, dump = 8 * 18432 + wid * 256 + lane * 4;
#define CWR(m) { unsigned a = off[m] * 4; \
        _Pragma("unroll") for (int r = 0; r < 4; ++r) _Pragma("unroll") for (int n = 0; n < 4; ++n) { \
          const bool neg = !(acc[m][n][r] > thr); \
          *(LAS float*)(lds + (neg ? dump : (m >= 4 ? min(a, a - 18432u) : a) + lb)) = acc[m][n][r]; a += neg ? 0u : 4u; } }
#define CRD(c) { const unsigned L = min(ng - c * 64u, 64u) * 16u; \
        _Pragma("unroll") for (int w = 0; w < 8; ++w) { const unsigned a = c * 8192 + w * L + lane * 16; cf[c][w] = *(const LAS f32x4*)(lw + (c == 2 ? min(a, a - 18432u) : a)); } }
#pragma unroll
      for (int c = 0; c < 3; ++c)
#pragma unroll
        for (int w = 0; w < 8; ++w) cf[c][w] = (f32x4){0.f, 0.f, 0.f, 0.f};
      CWR(0) CWR(1) CWR(2) CWR(3)
      FENCE();
      if (2048u <= cnt1) CRD(0)
      FENCE();
      CWR(4) CWR(5)
      if (cnt + lane < ng * 32u) { const unsigned a = (cnt + lane) * 4; *(LAS float*)(lw + min(a, a - 18432u)) = 0.f; }
      FENCE();
      if (2048u > cnt1) CRD(0)
      if (2048u < cnt) CRD(1)
      if (4096u < cnt) CRD(2)
      FENCE();
      *(LAS u32x4*)(lw + lane * 16) = (u32x4){0u, 0u, 0u, 0u};
    }
    if (wid == 0) {
      const u64 t1 = NOW(); u32x2 gv;
      for (;;) {
        gv = __builtin_amdgcn_raw_buffer_load_b64(rl, lane == 0 ? OFF_AMF + (b & 7) * 256 : OFF_TMO, 0, AUX_AG); FENCE();
        if ((unsigned)uni((int)gv[1]) == call) break;
        __builtin_amdgcn_s_sleep(4);
        if (give_up(wsl, call, t1)) { gv[0] = call; break; }
      }
      if (lane < 2) red[8 + lane] = (int)gv[0];
      for (int x = 0; x < pm; ++x) __builtin_amdgcn_s_sleep(8);
    }
    BAR();
    const float amax = __int_as_float(red[8]); bad = (unsigned)red[9] == call;
    const float inv = amax > 0.f ? 127.f / amax : 0.f;
#define QM2(a, b) ({ const f32x2 t_ = __builtin_elementwise_fma((f32x2){a, b}, (f32x2){inv, inv}, (f32x2){8388608.f, 8388608.f}); __builtin_amdgcn_perm(__float_as_uint(t_[1]), __float_as_uint(t_[0]), 0x0c0c0400u); })
#define QUANTP(a, b, c, d) __builtin_amdgcn_perm(QM2(c, d), QM2(a, b), 0x05040100u)
#define QUANT4(a, b, c, d) QUANTP(fmaxf(a, 0.f), fmaxf(b, 0.f), fmaxf(c, 0.f), fmaxf(d, 0.f))
    {
      const unsigned so = OFF_EXV + (unsigned)sub * XVS;
      unsigned qq[3][8], kc[4] = {0u, 0u, 0u, 0u}, po[4], big = 0;
      u64 st[3];
#pragma unroll
      for (int c = 0; c < 3; ++c) {
        st[c] = 0;
#pragma unroll
        for (int h = 0; h < 8; ++h) qq[c][h] = 0;
        if (c * 64u + lane < ng) {
          unsigned m[8], L[8], B = 0;
#pragma unroll
          for (int j = 0; j < 8; ++j) {
            const unsigned q = qq[c][j] = QUANTP(cf[c][j][0], cf[c][j][1], cf[c][j][2], cf[c][j][3]);
            const unsigned u = q + 0x68686868u, g = u & 0x80808080u, g7 = g >> 7, k7 = g - g7;
            m[j] = q | (k7 & 0x18181818u);
            L[j] = (((u & k7) >> 3) & 0x0f0f0f0fu) + g7;
            B |= (j <= 4 ? m[j] >> (4 - j) : m[j] << (j - 4)) & (0x01010101u << j);
          }
          const u32x4 o = {BFI(0x0f0f0f0fu, m[0], m[4] << 4), BFI(0x0f0f0f0fu, m[1], m[5] << 4), BFI(0x0f0f0f0fu, m[2], m[6] << 4), BFI(0x0f0f0f0fu, m[3], m[7] << 4)};
          ST128(o, rl, lane * 16 + c * 1024, so, AUX_SYS);
          *(LAS unsigned*)(lw + (c * 64 + lane) * 4) = B;
#pragma unroll
          for (int h = 0; h < 4; ++h) {
            unsigned p0 = L[2 * h] + (L[2 * h] << 8); p0 += p0 << 16;
            unsigned p1 = L[2 * h + 1] + (p0 >> 24); p1 += p1 << 8; p1 += p1 << 16;
            const unsigned sp = ((1u << (p0 & 31u)) | (1u << ((p0 >> 8) & 31u)) | (1u << ((p0 >> 16) & 31u)) | (1u << ((p0 >> 24) & 31u)) |
                                 (1u << (p1 & 31u)) | (1u << ((p1 >> 8) & 31u)) | (1u << ((p1 >> 16) & 31u)) | (1u << ((p1 >> 24) & 31u))) >> 1;
            st[c] |= (u64)sp << (kc[c] & 63u);
            kc[c] += p1 >> 24;
            big |= p1;
          }
          big |= (kc[c] + 0x1fffffbfu);
          kc[3] += __builtin_popcountll(st[c]);
        }
      }
      const unsigned tot = chunk_offsets(kc, po), nb = (unsigned)uni((int)po[3]), nd = (nb + 31u) >> 5;
      const bool dense = __ballot(big >= 0x20000000u) || nd > 128u || ng + nd > 254u || 5u * ng + nd > 1024u || tot - nb > 2040u;
      if (!dense) {
#pragma unroll
        for (int c = 0; c < 3; ++c) {
          if (kc[c]) {
            const unsigned ad = (unsigned)(size_t)lw + (ng + (po[c] >> 5)) * 4u, sh = po[c] & 31u;
            const u64 sv = st[c] << sh;
            asm volatile("ds_or_b32 %0, %1\n\tds_or_b32 %0, %2 offset:4" :: "v"(ad), "v"((unsigned)sv), "v"((unsigned)(sv >> 32)) : "memory");
            if (sh + kc[c] > 64u) asm volatile("ds_or_b32 %0, %1 offset:8" :: "v"(ad), "v"((unsigned)(st[c] >> 32) >> (32u - sh)) : "memory");
          }
        }
      } else {
        u32x4 hp[2] = {{0u, 0u, 0u, 0u}, {0u, 0u, 0u, 0u}};
        WAIT_V0();
#pragma unroll
        for (int c = 0; c < 3; ++c) {
          if (c * 64u + lane < ng) {
            unsigned m[8], B = 0, B5 = 0, B6 = 0;
#pragma unroll
            for (int j = 0; j < 8; ++j) {
              m[j] = qq[c][j];
              B |= ((m[j] >> 4) & 0x01010101u) << j; B5 |= ((m[j] >> 5) & 0x01010101u) << j; B6 |= ((m[j] >> 6) & 0x01010101u) << j;
            }
            const u32x4 o = {BFI(0x0f0f0f0fu, m[0], m[4] << 4), BFI(0x0f0f0f0fu, m[1], m[5] << 4), BFI(0x0f0f0f0fu, m[2], m[6] << 4), BFI(0x0f0f0f0fu, m[3], m[7] << 4)};
            ST128(o, rl, lane * 16 + c * 1024, so, AUX_SYS);
            *(LAS unsigned*)(lw + (c * 64 + lane) * 4) = B;
            hp[c >> 1][(c & 1) * 2] = B5; hp[c >> 1][(c & 1) * 2 + 1] = B6;
          }
        }
        ST128(hp[0], rl, lane * 16, OFF_EXD + (unsigned)sub * 2048u, AUX_SYS); ST128(hp[1], rl, lane * 16 + 1024, OFF_EXD + (unsigned)sub * 2048u, AUX_SYS);
      }
      FENCE();
      const unsigned td = dense ? ng : ng + nd;
      if (lane * 4u < td) ST128(*(const LAS u32x4*)(lw + lane * 16), rl, lane * 16, so + ng * 16u, AUX_SYS);
      const unsigned tw = 4u * ng + td, r = tw & 31u, u = !dense && pm >= TR0 && tw >= 32u && r <= td && r <= 4u * TUM ? (r + 3u) >> 2 : 0u;
      if (lane < u) *(LAS u32x4*)(lds + 8 * 18432 + wid * 128 + lane * 16) = *(const LAS u32x4*)(lw + (td - r) * 4u + lane * 16);
      if (lane == 0) {
        *(volatile LAS unsigned char*)(lds + LDS_CTL + 128 + wid) = (unsigned char)(((u ? tw : tw + 31u) >> 5) | (dense ? 0x80u : 0u));
        *(volatile LAS unsigned char*)(lds + LDS_CTL + 136 + wid) = (unsigned char)u;
      }
    }
    WAIT_V0(); BAR();
    if (pm < TR0) {
      if (tid < 2 * NR && (tid >> 1) != rank)
        st_sys64(pw + OFF_VFLAG + rank * CSTR + tile * 16 + (tid & 1) * 8, ((u64)*(volatile LAS unsigned*)(lds + LDS_CTL + 128 + (tid & 1) * 4) << 32) | 0xffffff80u | call);
    } else if (wid == 7) {
      const unsigned uj = *(volatile LAS unsigned char*)(lds + LDS_CTL + 136 + (lane >> 3));
      const u64 on = __ballot((unsigned)(lane & 7) < uj);
      const unsigned at = __builtin_amdgcn_mbcnt_hi((unsigned)(on >> 32), __builtin_amdgcn_mbcnt_lo((unsigned)on, 0u));
      if (!(lane & 7)) *(volatile LAS unsigned char*)(lds + LDS_CTL + 144 + (lane >> 3)) = (unsigned char)(uj ? at : 63u);
      if (on) {
        const u32x4 d = *(const LAS u32x4*)(lds + 8 * 18432 + lane * 16);
        if ((unsigned)(lane & 7) < uj) *(LAS u32x4*)(lds + 8 * 18432 + 1024 + at * 16) = d;
        FENCE();
        const u32x4 e = *(const LAS u32x4*)(lds + 8 * 18432 + 1024 + lane * 16);
        const unsigned oo = OFF_VT + (unsigned)(rank * NGEMM + tile) * 1024u, oe = oo + ((__builtin_popcountll(on) + 3u) >> 2) * 64u;
#pragma unroll
        for (int p = 1; p < NR; ++p) {
          const int pd = (rank + p) & 7;
          const u64 base = ((u64)(unsigned)__builtin_amdgcn_readlane((int)((u64)pw >> 32), 2 * pd) << 32) | (unsigned)__builtin_amdgcn_readlane((int)(u64)pw, 2 * pd);
          ST128(e, mk_rsrc((const void*)base, oe), lane * 16, oo, AUX_SYS);
        }
        WAIT_V0();
      }
      const unsigned ub = *(volatile LAS unsigned*)(lds + LDS_CTL + 144 + (lane & 1) * 4), u24 = ((ub & 63u) << 8) | ((ub & 0x3f00u) << 6) | ((ub & 0x3f0000u) << 4) | ((ub & 0x3f000000u) << 2);
      if (lane < 2 * NR && (lane >> 1) != rank)
        st_sys64(pw + OFF_VFLAG + rank * CSTR + tile * 16 + (lane & 1) * 8, ((u64)*(volatile LAS unsigned*)(lds + LDS_CTL + 128 + (lane & 1) * 4) << 32) | u24 | (call & 0x7fu) | 0x80u);
    }
    {
      __amdgpu_buffer_rsrc_t rd = mk_rsrc(P.out, OUT_BYTES);
      const unsigned ovoff = (unsigned)(wr * 96 + (lane & 15)) * GN + wc * 64 + (lane >> 4) * 16;
      const unsigned osoff = (unsigned)(pm * 192) * GN + rank * NLOC + pn * 256;
      if (dw == 1 << 20) { __builtin_amdgcn_s_sleep(100); __builtin_amdgcn_s_sleep(100); }
#pragma unroll
      for (int m = 0; m < 6; ++m) {
        u32x4 q;
#pragma unroll
        for (int r = 0; r < 4; ++r) q[r] = QUANT4(acc[m][0][r], acc[m][1][r], acc[m][2][r], acc[m][3][r]);
        if (bad) q[0] = 0x80808080u;
        ST128(q, rd, ovoff + m * 16 * GN, osoff, AUX_OUT);
      }
    }
  }
  {
    const bool late = b >= GEMM0;
    if (dw < (1 << 20)) {
      const unsigned call = (unsigned)uni((int)prev) + 1u;
      const int rel = uni(dw % 7), k = uni(dw / 7);
      const int src = (rank + 1 + rel) & 7;
      char* wsr = P.ws[src];
      __amdgpu_buffer_rsrc_t rl = mk_rsrc(wsl, WS_NEED);
      LAS char* sb = lds + (late ? wid * 18432 : (7 - wid) * 11264);
      *(LAS u32x4*)(sb + 9760 + lane * 16) = *(const GAS u32x4*)((const GAS char*)&RUNS + lane * 16);
      const unsigned tbase = (call - 1u) * (unsigned)(NSUB + DPG) - (unsigned)DPL;
      char* tp = wsl + OFF_VTICK + rel * CSTR;
      const unsigned fo = OFF_VFLAG + src * CSTR, vt0 = OFF_VT + (unsigned)src * NGEMM * 1024u;
      const unsigned mwant = call * (lane == 21 ? 8u : 16u);
      u64 rdy = 0, mrdy = (1u << PMP) - 1u;
      u32x4 fl = {0, 0, 0, 0}; unsigned mc = 0, rb = 0, rowlim = 0;
      u32x4 v[4] = {}, dv[2] = {}; u32x3 mkn = {};
      unsigned tk = 0, n = (unsigned)k;
      int cur = -1; unsigned cur_t = 0; bool cur_dn = false, more = true, polled = false;
      const u64 t0 = NOW();
      const unsigned ovoff = (unsigned)(lane & 15) * GN + (lane >> 4) * 16;
      bad = uni(bad) != 0;
      if (late) n = (unsigned)uni(__builtin_amdgcn_raw_ptr_buffer_atomic_add_i32(1, mk_rsrc(tp, 4u), lane * 4, 0, AUX_AG)) - tbase;
      else {
        while (ld_ag(wsl + OFF_AMF + (8 + (b & 7)) * 256 + 4) != call) {
          __builtin_amdgcn_s_sleep(20);
          if (give_up(wsl, call, t0)) break;
        }
        __builtin_amdgcn_s_sleep(40);
      }
      WAIT_V0();
      do {
        if (polled && !bad) {
          rdy = __ballot((((fl[0] ^ (call | 0x80u)) | (fl[2] ^ (call | 0x80u))) & 0xffu) == 0u); mrdy = __ballot(mc == mwant) | ((1u << PMP) - 1u); polled = false;
          rowlim = rb + min(~rdy ? (unsigned)__builtin_ctzll(~rdy) >> 3 : 8u, (unsigned)__builtin_ctzll(~(mrdy >> rb)));
        }
        const u32x3 mk = mkn;
        unsigned off[6] = {0u, 0u, 0u, 0u, 0u, 0u};
        if (cur >= 0) {
          const u32x4 (&o)[4] = v; const u32x4 (&dn)[2] = dv;
          const unsigned ng = (piece_offsets(mk, off) + 31u) >> 5;
#pragma unroll
          for (int c = 0; c < 4; ++c) {
            const unsigned x = c * 64u + lane - ng;
            if (((c == 1 || c == 2) || (c == 0 ? ng < 64u : ng > 128u)) && x < 64u) *(LAS u32x4*)(sb + 6400 + x * 16) = o[c];
          }
          FENCE();
          if (cur_t && lane < TUM) *(LAS u32x4*)(sb + 6400 + (((cur_t - ng) & 63u) + lane) * 16) = dn[0];
          *(LAS u32x4*)(sb + 7680 + lane * 16) = (u32x4){0u, 0u, 0u, 0u};
          FENCE();
          unsigned B[3], F[3] = {0u, 0u, 0u}, kc[4], po[4], t[4];
#pragma unroll
          for (int c = 0; c < 3; ++c) B[c] = *(const LAS unsigned*)(sb + 6400 + (c * 64 + lane) * 4);
          unsigned sw = *(const LAS unsigned*)(sb + 6400 + (ng + lane) * 4);
          const unsigned sw2 = *(const LAS unsigned*)(sb + 6656 + (ng + lane) * 4);
          FENCE();
#define RUNS_GET() { _Pragma("unroll") for (int z = 0; z < 4; ++z) t[z] = *(const LAS unsigned*)(sb + 9760 + ((sw >> (8 * z)) & 0xffu) * 4); FENCE(); }
          RUNS_GET()
#pragma unroll
          for (int c = 0; c < 3; ++c) {
            if (c < 2 || ng > 128u) {
              const unsigned T = ((o[c][0] >> 3) & 0x11111111u) | ((o[c][1] >> 2) & 0x22222222u) | ((o[c][2] >> 1) & 0x44444444u) | (o[c][3] & 0x88888888u);
              F[c] = B[c] & T & (unsigned)((int)(c * 64u + lane - (cur_dn ? 0u : ng)) >> 31);
            }
            kc[c] = __builtin_popcount(F[c]);
          }
          kc[3] = __builtin_popcount(sw);
          const unsigned xb = ((unsigned)(size_t)sb + 7680u) * 2u;
          const unsigned tot = chunk_offsets(kc, po), ne = (unsigned)uni((int)po[3]);
          unsigned zo = po[3] + (xb - ne);
          RUNS_PUT()
          if (tot - ne < ne) {
            sw = sw2;
            RUNS_GET()
            const unsigned k2 = __builtin_popcount(sw);
            zo = (tot - ne + xb) + scan_incl(k2) - k2;
            RUNS_PUT()
          }
          FENCE();
          unsigned eo[3][4]; u32x2 sv[3][4];
#define EXT_GET(c) { _Pragma("unroll") for (int h = 0; h < 4; ++h) eo[c][h] = h ? eo[c][h ? h - 1 : 0] + __builtin_popcount(F[c] & (0x03030303u << (2 * h - 2))) : po[c] + xb; \
            _Pragma("unroll") for (int h = 0; h < 4; ++h) sv[c][h] = *(const LAS u32x2a4*)(size_t)((eo[c][h] >> 1) & ~3u); }
          EXT_GET(0) EXT_GET(1)
          FENCE(); SCHED();
#pragma unroll
          for (int c = 0; c < 3; ++c) {
            if (c * 64u + lane < ng) {
              unsigned st[4];
              if (c == 2) { EXT_GET(2) SCHED(); }
#pragma unroll
              for (int h = 0; h < 4; ++h) st[h] = __builtin_amdgcn_alignbit(sv[c][h][1], sv[c][h][0], eo[c][h] * 4u);
              const unsigned L = min(ng - c * 64u, 64u) * 4u;
              unsigned wa = (unsigned)(size_t)sb + lane * 4;
#pragma unroll
              for (int q = 0; q < 8; ++q) {
                const unsigned g = (q >= 2 ? F[c] >> (q - 2) : F[c] << (2 - q)) & 0x04040404u;
                unsigned y = __builtin_amdgcn_perm(0u, st[q >> 1], 0x0c010c00u);
                y = (y | (y << 4)) & 0x0f0f0f0fu;
                if (!(q & 1)) st[q >> 1] >>= 4u * __builtin_popcount(g);
                unsigned d = BFI(0x0f0f0f0fu, q < 4 ? o[c][q] : o[c][q & 3] >> 4, q <= 4 ? B[c] << (4 - q) : B[c] >> (q - 4)) & 0x1f1f1f1fu;
                d += __builtin_amdgcn_perm(y, 0u, GSEL(g)) << 3;
                *(LAS unsigned*)(size_t)(wa + c * 2048) = d;
                wa += L; asm volatile("" : "+v"(wa));
              }
              if (cur_dn) {
#pragma unroll
                for (int q = 7; q >= 0; --q) {
                  wa -= L;
                  asm volatile("ds_or_b32 %0, %1" :: "v"(wa + c * 2048), "v"((((dn[c >> 1][(c & 1) * 2] >> q) & 0x01010101u) << 5) | (((dn[c >> 1][(c & 1) * 2 + 1] >> q) & 0x01010101u) << 6)) : "memory");
                }
              }
            }
          }
        }
        FENCE();
        int nxt = -1; bool nxt_dn = false; unsigned lim = 0, nxt_t = 0, st = 0;
        if (more) {
          if (n < (unsigned)NSUB) {
            const unsigned row = n >> 6, li = (row - rb) * 8u + ((n & 31u) >> 2), pa = min(row + 2u, 21u);
            const bool watch = pa >= rowlim;
            if (!watch || (li < 64u && ((rdy >> li) & (mrdy >> row) & 1))) {
              nxt = (int)n;
              const unsigned f0 = n & 32u ? fl[2] : fl[0], f1 = n & 32u ? fl[3] : fl[1];
              const unsigned info = ((unsigned)__builtin_amdgcn_readlane((int)f1, li & 63u) >> (8 * (n & 3))) & 0xffu, at = ((unsigned)__builtin_amdgcn_readlane((int)f0, li & 63u) >> (8 + 6 * (n & 3))) & 63u;
              nxt_dn = info >= 128u;
              lim = OFF_EXV + n * XVS + (info & 63u) * 128u;
              st = vt0 + (row * 8u + ((n & 31u) >> 2)) * 1024u + at * 16u;
              nxt_t = at == 63u ? 0u : (info & 63u) * 8u;
            }
            if (watch) {
              const bool wide = n >= (unsigned)DPL || nxt >= 0;
              rb = row;
              fl = __builtin_amdgcn_raw_buffer_load_b128(mk_rsrc(wsl, fo + row * 128u + (!wide ? 128u : n < 128u ? 512u : 1024u)), lane * 16, fo + row * 128u, AUX_SYS);
              if (!((mrdy >> min(row + 7u, 21u)) & (mrdy >> row) & 1)) mc = __builtin_amdgcn_raw_buffer_load_b32(rl, lane * 4, OFF_MRC + src * CSTR, AUX_SYS);
              polled = true;
            }
          } else more = false;
        }
        {
          const unsigned j = nxt >= 0 ? n : 0u;
          __amdgpu_buffer_rsrc_t rv = mk_rsrc(wsr, lim);
#pragma unroll
          for (int c = 0; c < 4; ++c) v[c] = __builtin_amdgcn_raw_buffer_load_b128(rv, lane * 16 + c * 1024, OFF_EXV + j * XVS, AUX_SYS);
          dv[0] = __builtin_amdgcn_raw_buffer_load_b128(mk_rsrc(nxt_dn ? wsr : wsl, nxt_dn ? WS_NEED : nxt_t ? st + TUM * 16u : 0u), lane * 16, nxt_dn ? OFF_EXD + j * 2048u : st, AUX_SYS);
          dv[1] = __builtin_amdgcn_raw_buffer_load_b128(mk_rsrc(wsr, nxt_dn ? WS_NEED : 0u), lane * 16 + 1024, OFF_EXD + j * 2048u, AUX_SYS);
          mkn = __builtin_amdgcn_raw_buffer_load_b96(mk_rsrc(wsl, nxt >= 0 ? WS_NEED : 0u), lane * 12, OFF_LM + ((unsigned)src * NSUB + j) * 768u, AUX_SYS);
        }
        tk = (unsigned)__builtin_amdgcn_raw_ptr_buffer_atomic_add_i32(1, mk_rsrc(tp, nxt >= 0 ? 4u : 0u), lane * 4, 0, AUX_AG);
        FENCE();
        u32x4 q[6] = {};
        if (cur >= 0) {
          unsigned ga[24], gs[24];
#pragma unroll
          for (int m = 0; m < 6; ++m) {
            unsigned a = off[m] + (unsigned)(size_t)sb;
#pragma unroll
            for (int r = 0; r < 4; ++r) {
              const int kk = (m & 1) * 4 + r;
              const unsigned g = (kk >= 2 ? mk[m >> 1] >> (kk - 2) : mk[m >> 1] << (2 - kk)) & 0x04040404u;
              ga[m * 4 + r] = a; gs[m * 4 + r] = g;
              a += __builtin_popcount(g);
            }
          }
          u32x2 sv[24];
#pragma unroll
          for (int e = 0; e < 24; ++e) sv[e] = *(const LAS u32x2a4*)(size_t)(ga[e] & ~3u);
          SCHED();
#pragma unroll
          for (int e = 0; e < 24; ++e) q[e >> 2][e & 3] = __builtin_amdgcn_perm(__builtin_amdgcn_alignbyte(sv[e][1], sv[e][0], ga[e]), 0u, GSEL(gs[e]));
          if (bad) q[0][0] = 0x80808080u;
        } else if (nxt < 0 && more) {
          __builtin_amdgcn_s_sleep(6);
          if (give_up(wsl, call, t0)) { rowlim = 22; bad = true; }
        }
        FENCE();
        {
          __amdgpu_buffer_rsrc_t rd = mk_rsrc(P.out, cur >= 0 ? OUT_BYTES : 0u);
          const unsigned qoff = cur >= 0 ? (unsigned)((cur >> 5) * 96) * GN + src * NLOC + (cur & 31) * 64 : 0u;
#pragma unroll
          for (int m = 0; m < 6; ++m) ST128(q[m], rd, ovoff + m * 16 * GN, qoff, AUX_OUT);
        }
        cur = nxt; cur_dn = nxt_dn; cur_t = nxt_t;
        WAIT_V(6); __builtin_amdgcn_s_waitcnt(0x0F76);
        asm volatile("" : "+v"(tk), "+v"(mc), "+v"(fl), "+v"(mkn), "+v"(dv[0]), "+v"(dv[1]), "+v"(v[0]), "+v"(v[1]), "+v"(v[2]), "+v"(v[3]));
        if (cur >= 0) n = (unsigned)uni((int)tk) - tbase;
      } while (more);
    }
  }
  __syncthreads();
  if (tid == 0) st_ag(wsl + OFF_CALL + b * 64, prev + 1u);
}
}

extern "C" void kernel_launch(int rank, int nranks, ncclComm_t comm, hipStream_t stream,
                              void* const* d_in, const int* in_sizes, int n_in,
                              void* d_out, int out_size, const int* in_dtypes, int out_dtype,
                              void* const* d_out_all, void* const* d_ws_all, size_t ws_bytes) {
  if (nranks != NR || (unsigned)rank >= (unsigned)NR || ws_bytes < WS_NEED || out_size != GM * GN) { fprintf(stderr, "unexpected configuration\n"); return; }
  Params P;
  P.x = (const float*)d_in[0]; P.w = (const float*)d_in[1]; P.out = (char*)d_out;
  const size_t top = (ws_bytes - WS_NEED) & ~(size_t)(2 * MiB - 1);
  for (int p = 0; p < NR; ++p) P.ws[p] = (char*)d_ws_all[p] + top;
  P.rank = rank; P.pad = 0; P.wsl = P.ws[rank];
  hipLaunchKernelGGL(fused_kernel, dim3(NWG), dim3(512), 0, stream, P);
}
```
